```python
import numpy as np

kernel_name = "gpu_gemm_ar_m4096_k4096_n16384_bf16_qe4m3"


ABSMAX_RTOL = 0.09

COMPUTE_DTYPE = "bfloat16"
EPILOGUE = "quant_e4m3"
IN_QUANT = None
OUT_QUANT = "e4m3"
IN_DTYPES = {"x": "bfloat16", "w_mat": "bfloat16"}
WORLD_SIZE = 8

def _to_bf16(x):
    u = np.ascontiguousarray(x, dtype=np.float32).view(np.uint32)
    return ((u + 0x7FFF + ((u >> 16) & 1)) & 0xFFFF0000).view(np.float32)

def setup_inputs():

    x = _to_bf16(np.random.randn(4096, 4096).astype(np.float32))
    w_mat = _to_bf16((np.random.randn(4096, 16384) / np.sqrt(4096)).astype(np.float32))
    return {"x": x, "w_mat": w_mat}

def _snap_fp8(x, mant_bits, max_abs, min_normal):
    a = np.abs(x.astype(np.float64))
    _m, _e = np.frexp(a)
    q = np.where(a >= min_normal, np.ldexp(1.0, _e - 1 - mant_bits),
                 min_normal * 2.0 ** -mant_bits)
    snapped = np.minimum(np.round(a / q) * q, max_abs)
    return (np.sign(x) * snapped).astype(np.float32)
def _quant_dequant(y, amax):
    scale = amax / 448.0
    q = _snap_fp8(y / scale, 3, 448.0, 2.0 ** -6)
    return (q * scale).astype(np.float32)
def _epilogue(y):
    return _quant_dequant(y, np.abs(y).max())

def reference(x, w_mat):
    y = (x @ w_mat).astype(np.float32)
    return _epilogue(y)
```

```cpp
#include <hip/hip_runtime.h>
#include <rccl/rccl.h>
#include <cstdio>
#include <cstdint>

namespace {

constexpr int NR = 8;
constexpr int MDIM = 4096, KSH = 512, NDIM = 16384, NBLK = 2048;
constexpr int NWG = 256;
constexpr int NTHREADS = 640;
constexpr int NPULL_PER_PEER = 16;
constexpr int NPULLERS = 7 * NPULL_PER_PEER;
constexpr int REDUCER_WG = 255;
constexpr unsigned long long DEADLINE_TICKS = 100000000ull;

constexpr unsigned STG_X = 0;
constexpr unsigned STG_W = 32u << 20;
constexpr unsigned EXP_W = 48u << 20;
constexpr unsigned EXP_O = 64u << 20;
constexpr unsigned FLG = 72u << 20;
constexpr unsigned FX_SRC = FLG;
constexpr unsigned FX_RDY = FLG + (32u << 10);
constexpr unsigned FW_SRC = FLG + (64u << 10);
constexpr unsigned FW_RDY = FLG + (80u << 10);
constexpr unsigned FO_SRC = FLG + (96u << 10);
constexpr unsigned AMAX_WG = FLG + (160u << 10);
constexpr unsigned AMAX_G = FLG + (164u << 10);
constexpr unsigned EPOCH = FLG + (168u << 10);
constexpr unsigned TMO = FLG + (168u << 10) + 256;
constexpr unsigned DBG = FLG + (256u << 10);
constexpr unsigned WS_NEED = FLG + (1u << 20);

constexpr int STAGE_BYTES = 48 * 1024;
constexpr int L_CTRL = 3 * STAGE_BYTES;
constexpr int L_QUEUE = L_CTRL;
constexpr int L_BAR = L_CTRL + 256;
constexpr int L_AMAX = L_CTRL + 260;
constexpr int L_ACNT = L_CTRL + 264;
constexpr int L_SCALE = L_CTRL + 268;
constexpr int L_SFLAG = L_CTRL + 272;
constexpr int LDS_BYTES = L_CTRL + 512;

typedef __attribute__((ext_vector_type(8))) short bf16x8;
typedef __attribute__((ext_vector_type(4))) float f32x4;
typedef __attribute__((ext_vector_type(4))) unsigned u32x4;
typedef __attribute__((ext_vector_type(2))) unsigned u32x2;
typedef __attribute__((address_space(3))) unsigned lds_u32;
typedef __attribute__((address_space(1))) unsigned glb_u32;

struct Params {
  const char* x;
  const char* w;
  char* out;
  char* ws[NR];
  int rank;
  unsigned ws_bytes;
};

constexpr int AUX_SC1 = 16, AUX_SYS = 17;

__device__ __forceinline__ __amdgpu_buffer_rsrc_t mk_rsrc(const void* p, unsigned bytes) {
  return __builtin_amdgcn_make_buffer_rsrc(const_cast<void*>(p), 0, bytes, 0x00020000);
}
template <int N> __device__ __forceinline__ void wait_vm() { asm volatile("s_waitcnt vmcnt(%0)" ::"i"(N) : "memory"); }
__device__ __forceinline__ unsigned rfl(unsigned v) { return __builtin_amdgcn_readfirstlane(v); }
__device__ __forceinline__ unsigned lds_ld(const char* smem, int off) {
  return rfl(__hip_atomic_load((const lds_u32*)(smem + off), __ATOMIC_RELAXED, __HIP_MEMORY_SCOPE_WORKGROUP));
}
__device__ __forceinline__ void lds_st(char* smem, int off, unsigned v) {
  __hip_atomic_store((lds_u32*)(smem + off), v, __ATOMIC_RELAXED, __HIP_MEMORY_SCOPE_WORKGROUP);
}
__device__ __forceinline__ unsigned long long now() { return __builtin_amdgcn_s_memrealtime(); }

#define STAMP(slot) do {} while (0)

__device__ __forceinline__ bool give_up(__amdgpu_buffer_rsrc_t self, unsigned seq, unsigned long long t0) {
  unsigned t = rfl(__builtin_amdgcn_raw_buffer_load_b32(self, TMO, 0, AUX_SYS));
  if (t == seq || now() - t0 > DEADLINE_TICKS) {
    __builtin_amdgcn_raw_buffer_store_b32(seq, self, TMO, 0, AUX_SYS);
    return true;
  }
  return false;
}

__device__ __forceinline__ bool wait_flag(__amdgpu_buffer_rsrc_t self, unsigned off, unsigned seq) {
  unsigned long long t0 = now();
  for (unsigned spins = 0;; ++spins) {
    asm volatile("" ::: "memory");
    unsigned v = rfl(__builtin_amdgcn_raw_buffer_load_b32(self, off, 0, AUX_SYS));
    if (v >= seq) return true;
    if ((spins & 63) == 63 && give_up(self, seq, t0)) return false;
    __builtin_amdgcn_s_sleep(1);
  }
}

__device__ __forceinline__ void glds4(unsigned voff, const char* sbase, unsigned lds_dst) {
  unsigned keep;
  asm volatile(
      "s_mov_b32 %0, m0\n\ts_mov_b32 m0, %3\n\ts_nop 0\n\t"
      "global_load_lds_dwordx4 %1, %2 sc1\n\t"
      "global_load_lds_dwordx4 %1, %2 offset:1024 sc1\n\t"
      "global_load_lds_dwordx4 %1, %2 offset:2048 sc1\n\t"
      "global_load_lds_dwordx4 %1, %2 offset:3072 sc1\n\t"
      "s_mov_b32 m0, %0"
      : "=&s"(keep) : "v"(voff), "s"(sbase), "s"(lds_dst) : "memory");
}
__device__ __forceinline__ void glds2(unsigned voff, const char* sbase, unsigned lds_dst) {
  unsigned keep;
  asm volatile(
      "s_mov_b32 %0, m0\n\ts_mov_b32 m0, %3\n\ts_nop 0\n\t"
      "global_load_lds_dwordx4 %1, %2 sc1\n\t"
      "global_load_lds_dwordx4 %1, %2 offset:1024 sc1\n\t"
      "s_mov_b32 m0, %0"
      : "=&s"(keep) : "v"(voff), "s"(sbase), "s"(lds_dst) : "memory");
}

__device__ __forceinline__ void export_job(const Params& P, unsigned n, unsigned lane, __amdgpu_buffer_rsrc_t self,
                                           unsigned& seq, bool first) {
  const unsigned rank = P.rank;
  const unsigned s = n / 384, r = n % 384;
  if (r < 256) {
    const unsigned d = r & 7, j = r >> 4, h = (r >> 3) & 1;
    __amdgpu_buffer_rsrc_t rw = mk_rsrc(P.w, KSH * NDIM * 2);
    const unsigned o = lane & 15, g = lane >> 4;
    const unsigned voff = ((g * 8) * NDIM + o * 8) * 2;
    const unsigned soff = ((s * 64 + h * 32) * NDIM + d * NBLK + j * 128) * 2;
    u32x4 row[8];
#pragma unroll
    for (int jj = 0; jj < 8; ++jj) row[jj] = __builtin_amdgcn_raw_buffer_load_b128(rw, voff + jj * NDIM * 2, soff, 0);
    if (first) seq = rfl(__builtin_amdgcn_raw_buffer_load_b32(self, EPOCH, 0, AUX_SYS)) + 1;
    const unsigned dst = (d == rank ? STG_W + (rank * 8 + s) * 16 * 16384 : EXP_W + (d * 8 + s) * 16 * 16384) + j * 16384 + h * 8192;
#pragma unroll
    for (int t = 0; t < 8; ++t) {
      u32x4 f;
#pragma unroll
      for (int e = 0; e < 4; ++e)
        f[e] = __builtin_amdgcn_perm(row[2 * e + 1][t >> 1], row[2 * e][t >> 1], (t & 1) ? 0x07060302u : 0x05040100u);
      __builtin_amdgcn_raw_buffer_store_b128(f, self, lane * 16 + t * 1024, dst, AUX_SYS);
    }
    wait_vm<0>();
    if (d == rank) {
      if (lane == 0) {
        u32x2 v = {seq, seq};
        __builtin_amdgcn_raw_buffer_store_b64(v, self, FW_RDY + (((rank * 8 + s) * 16 + j) * 4 + h * 2) * 4, 0, AUX_SYS);
      }
    } else if (lane == 0) {
      __amdgpu_buffer_rsrc_t rp = mk_rsrc(P.ws[d], P.ws_bytes);
      __builtin_amdgcn_raw_buffer_store_b32(seq, rp, FW_SRC + (((rank * 8 + s) * 16 + j) * 2 + h) * 4, 0, AUX_SYS);
    }
  } else {
    const unsigned i = (r - 256) >> 3, sub = (r - 256) & 7;
    __amdgpu_buffer_rsrc_t rx = mk_rsrc(P.x, MDIM * KSH * 2);
    const unsigned c = lane & 15, q = lane >> 4;
    const unsigned voff = (c * KSH + q * 8) * 2;
    const unsigned soff = ((i * 256 + sub * 32) * KSH + s * 64) * 2;
    u32x4 f[4];
#pragma unroll
    for (int b = 0; b < 4; ++b)
      f[b] = __builtin_amdgcn_raw_buffer_load_b128(rx, voff + ((b >> 1) * 16 * KSH + (b & 1) * 32) * 2, soff, 0);
    if (first) seq = rfl(__builtin_amdgcn_raw_buffer_load_b32(self, EPOCH, 0, AUX_SYS)) + 1;
    const unsigned idx = ((rank * 8 + s) * 16 + i) * 8 + sub;
#pragma unroll
    for (int b = 0; b < 4; ++b) __builtin_amdgcn_raw_buffer_store_b128(f[b], self, lane * 16 + b * 1024, STG_X + idx * 4096, AUX_SYS);
    wait_vm<0>();
    if (lane < NR) {
      glb_u32* fp = (glb_u32*)(P.ws[lane] + (lane == rank ? FX_RDY : FX_SRC) + idx * 4);
      __hip_atomic_store(fp, seq, __ATOMIC_RELAXED, __HIP_MEMORY_SCOPE_SYSTEM);
    }
  }
}

__global__ void __launch_bounds__(NTHREADS, 3) fused_kernel(Params P) {
  __shared__ __attribute__((aligned(16))) unsigned smem32[LDS_BYTES / 4];
  char* const smem = (char*)smem32;
  const unsigned lane = threadIdx.x & 63;
  const unsigned wave = rfl(threadIdx.x >> 6);
  const unsigned b = blockIdx.x;
  const unsigned rank = P.rank;
  char* const wsl = P.ws[rank];
  __amdgpu_buffer_rsrc_t self = mk_rsrc(wsl, P.ws_bytes);
  const unsigned xg = b & 7, xi = b >> 3;
  const unsigned ti = (xg >> 1) * 4 + (xi >> 3), tj = (xg & 1) * 8 + (xi & 7);

  if (threadIdx.x < 128) *(unsigned*)(smem + L_CTRL + threadIdx.x * 4) = 0;
  __syncthreads();

  if (wave < 8) {
    STAMP(wave == 0 ? 0 : 63);
    unsigned seq = 0;
    export_job(P, wave * 256 + b, lane, self, seq, true);
    if (wave < 4) export_job(P, 2048 + wave * 256 + b, lane, self, seq, false);
    if (wave == 0) STAMP(1);

    f32x4 acc[8][2];
#pragma unroll
    for (int t = 0; t < 8; ++t) { acc[t][0] = f32x4{0.f, 0.f, 0.f, 0.f}; acc[t][1] = f32x4{0.f, 0.f, 0.f, 0.f}; }
    const unsigned lds0 = (unsigned)(uintptr_t)(lds_u32*)smem;
    const unsigned voff = lane * 16;
    const char* xsrc = wsl + STG_X + (ti * 8 + wave) * 4096;
    const char* wsrc = wsl + STG_W + tj * 16384 + wave * 2048;
    unsigned issued = 0, ibuf = 0;
    unsigned cbuf = 0;
    auto issue = [&](unsigned ent) {
      const unsigned sg = ent - 1;
      const unsigned l = lds0 + ibuf * STAGE_BYTES;
      glds2(voff, wsrc + (size_t)sg * (16 * 16384), l + wave * 2048);
      glds4(voff, xsrc + (size_t)sg * (128 * 4096), l + 16384 + wave * 4096);
      ++issued;
      ibuf = ibuf == 2 ? 0 : ibuf + 1;
    };
    for (unsigned u = 0; u < 64; ++u) {
      if (issued == u) {
        unsigned e;
        while ((e = lds_ld(smem, L_QUEUE + issued * 4)) == 0) __builtin_amdgcn_s_sleep(1);
        issue(e);
      }
      if (issued == u + 1 && issued < 64) {
        unsigned e = lds_ld(smem, L_QUEUE + issued * 4);
        if (e) issue(e);
      }
      if (issued == u + 1) wait_vm<0>(); else wait_vm<6>();
      if (lane == 0) __hip_atomic_fetch_add((lds_u32*)(smem + L_BAR), 1u, __ATOMIC_RELAXED, __HIP_MEMORY_SCOPE_WORKGROUP);
      const unsigned target = 8 * (u + 1);
      unsigned e0, e1;
      const unsigned qi = issued < 63 ? issued : 63;
      do {
        asm volatile("" ::: "memory");
        unsigned bv = __hip_atomic_load((const lds_u32*)(smem + L_BAR), __ATOMIC_RELAXED, __HIP_MEMORY_SCOPE_WORKGROUP);
        unsigned q0 = __hip_atomic_load((const lds_u32*)(smem32 + L_QUEUE / 4 + qi), __ATOMIC_RELAXED, __HIP_MEMORY_SCOPE_WORKGROUP);
        e0 = rfl(bv); e1 = rfl(q0);
      } while (e0 < target);
      asm volatile("" ::: "memory");
      if (issued < 64 && e1) {
        issue(e1);
        if (issued == u + 2 && issued < 64) {
          unsigned e = lds_ld(smem, L_QUEUE + issued * 4);
          if (e) issue(e);
        }
      }
      const char* wb = smem + cbuf * STAGE_BYTES + lane * 16;
      const char* xb = wb + 16384 + wave * 4096;
      bf16x8 xf[2][2], wf[2][8];
#pragma unroll
      for (int ks = 0; ks < 2; ++ks) {
        xf[ks][0] = *(const bf16x8*)(xb + ks * 1024);
        xf[ks][1] = *(const bf16x8*)(xb + 2048 + ks * 1024);
#pragma unroll
        for (int t = 0; t < 8; ++t) wf[ks][t] = *(const bf16x8*)(wb + ks * 8192 + t * 1024);
      }
      __builtin_amdgcn_sched_barrier(0);
#pragma unroll
      for (int ks = 0; ks < 2; ++ks)
#pragma unroll
        for (int t = 0; t < 8; ++t) {
          acc[t][0] = __builtin_amdgcn_mfma_f32_16x16x32_bf16(wf[ks][t], xf[ks][0], acc[t][0], 0, 0, 0);
          acc[t][1] = __builtin_amdgcn_mfma_f32_16x16x32_bf16(wf[ks][t], xf[ks][1], acc[t][1], 0, 0, 0);
        }
      __builtin_amdgcn_sched_barrier(0);
      asm volatile("" ::: "memory");
      cbuf = cbuf == 2 ? 0 : cbuf + 1;
    }
    if (wave == 0) STAMP(2);
    float mx = 0.f;
#pragma unroll
    for (int t = 0; t < 8; ++t)
#pragma unroll
      for (int mt = 0; mt < 2; ++mt)
#pragma unroll
        for (int r = 0; r < 4; ++r) mx = fmaxf(mx, fabsf(acc[t][mt][r]));
#pragma unroll
    for (int o = 32; o >= 1; o >>= 1) mx = fmaxf(mx, __shfl_xor(mx, o));
    if (lane == 0) {
      __hip_atomic_fetch_max((lds_u32*)(smem + L_AMAX), __float_as_uint(mx), __ATOMIC_RELAXED, __HIP_MEMORY_SCOPE_WORKGROUP);
      __hip_atomic_fetch_add((lds_u32*)(smem + L_ACNT), 1u, __ATOMIC_RELAXED, __HIP_MEMORY_SCOPE_WORKGROUP);
    }
    while (lds_ld(smem, L_SFLAG) == 0) __builtin_amdgcn_s_sleep(1);
    asm volatile("" ::: "memory");
    const float inv = __uint_as_float(lds_ld(smem, L_SCALE));
    if (wave == 0) STAMP(3);
    char* tb = smem + STAGE_BYTES + wave * 4096;
    const unsigned c = lane & 15, q = lane >> 4;
#pragma unroll
    for (int mt = 0; mt < 2; ++mt) {
      unsigned dw[8];
#pragma unroll
      for (int r = 0; r < 4; ++r)
#pragma unroll
        for (int hh = 0; hh < 2; ++hh) {
          float v0 = __builtin_amdgcn_fmed3f(acc[4 * hh + 0][mt][r] * inv, -448.f, 448.f);
          float v1 = __builtin_amdgcn_fmed3f(acc[4 * hh + 1][mt][r] * inv, -448.f, 448.f);
          float v2 = __builtin_amdgcn_fmed3f(acc[4 * hh + 2][mt][r] * inv, -448.f, 448.f);
          float v3 = __builtin_amdgcn_fmed3f(acc[4 * hh + 3][mt][r] * inv, -448.f, 448.f);
          int p = __builtin_amdgcn_cvt_pk_fp8_f32(v0, v1, 0, false);
          p = __builtin_amdgcn_cvt_pk_fp8_f32(v2, v3, p, true);
          dw[2 * r + hh] = (inv != inv) ? 0x7f7f7f7fu : (unsigned)p;
        }
      const unsigned row = mt * 16 + c;
      *(u32x4*)(tb + row * 128 + (((2 * q) ^ (row & 7)) * 16)) = u32x4{dw[0], dw[1], dw[2], dw[3]};
      *(u32x4*)(tb + row * 128 + (((2 * q + 1) ^ (row & 7)) * 16)) = u32x4{dw[4], dw[5], dw[6], dw[7]};
    }
    u32x4 ov[4];
#pragma unroll
    for (int it = 0; it < 4; ++it) {
      const unsigned row = it * 8 + (lane >> 3);
      ov[it] = *(const u32x4*)(tb + row * 128 + (((lane & 7) ^ (row & 7)) * 16));
    }
    const unsigned chunk = b * 8 + wave;
#pragma unroll
    for (int it = 0; it < 4; ++it) __builtin_amdgcn_raw_buffer_store_b128(ov[it], self, lane * 16 + it * 1024, EXP_O + chunk * 4096, AUX_SYS);
    wait_vm<0>();
    if (lane < NR && lane != rank) {
      glb_u32* fp = (glb_u32*)(P.ws[lane] + FO_SRC + (rank * 2048 + chunk) * 4);
      __hip_atomic_store(fp, seq, __ATOMIC_RELAXED, __HIP_MEMORY_SCOPE_SYSTEM);
    }
    __amdgpu_buffer_rsrc_t ro = mk_rsrc(P.out, (unsigned)MDIM * NDIM);
    const unsigned ovoff = (lane >> 3) * NDIM + (lane & 7) * 16;
    const unsigned osoff = (ti * 256 + wave * 32) * NDIM + rank * NBLK + tj * 128;
#pragma unroll
    for (int it = 0; it < 4; ++it) __builtin_amdgcn_raw_buffer_store_b128(ov[it], ro, ovoff + it * 8 * NDIM, osoff, 0);
    if (wave == 0) STAMP(4);
  } else if (wave == 8) {
    __builtin_amdgcn_s_setprio(2);
    const unsigned seq = rfl(__builtin_amdgcn_raw_buffer_load_b32(self, EPOCH, 0, AUX_SYS)) + 1;
    const unsigned p = lane & 7, kind = lane >> 3;
    unsigned s = 0, cnt = 0;
    bool dead = false;
    unsigned long long t0 = now();
    for (unsigned spins = 0; cnt < 64; ++spins) {
      asm volatile("" ::: "memory");
      bool ok = true;
      if (kind < 3 && s < 8 && !dead) {
        const unsigned off = kind < 2 ? FX_RDY + (((p * 8 + s) * 16 + ti) * 8 + kind * 4) * 4 : FW_RDY + (((p * 8 + s) * 16 + tj) * 4) * 4;
        u32x4 v = __builtin_amdgcn_raw_buffer_load_b128(self, off, 0, AUX_SYS);
        ok = v[0] >= seq && v[1] >= seq && v[2] >= seq && v[3] >= seq;
      }
      const unsigned long long bal = __ballot(ok);
      const unsigned long long live = __ballot(s < 8);
      unsigned rdy = (unsigned)(bal & (bal >> 8) & (bal >> 16) & live) & 0xffu;
      if (rdy) {
        const unsigned mine = (rdy >> p) & 1;
        const unsigned pos = cnt + __popc(rdy & ((1u << p) - 1));
        if (mine && kind == 0) lds_st(smem, L_QUEUE + pos * 4, p * 8 + s + 1);
        s += mine;
        cnt += __popc(rdy);
      } else {
        if ((spins & 63) == 63 && give_up(self, seq, t0)) dead = true;
        __builtin_amdgcn_s_sleep(2);
      }
    }
    STAMP(8);
    while (lds_ld(smem, L_ACNT) < 8) __builtin_amdgcn_s_sleep(1);
    asm volatile("" ::: "memory");
    const unsigned am = lds_ld(smem, L_AMAX);
    if (lane == 0) {
      u32x2 g = {am, seq};
      __builtin_amdgcn_raw_buffer_store_b64(g, self, AMAX_WG + b * 8, 0, AUX_SYS);
    }
    STAMP(9);
    unsigned gm = 0;
    t0 = now();
    for (unsigned spins = 0;; ++spins) {
      asm volatile("" ::: "memory");
      u32x2 g = __builtin_amdgcn_raw_buffer_load_b64(self, AMAX_G + (b & 7) * 128 + (lane & 7) * 8, 0, AUX_SYS);
      if (__all(g[1] >= seq)) { gm = g[0]; break; }
      if ((spins & 63) == 63 && give_up(self, seq, t0)) break;
      __builtin_amdgcn_s_sleep(1);
    }
#pragma unroll
    for (int o = 4; o >= 1; o >>= 1) gm = max(gm, (unsigned)__shfl_xor((int)gm, o));
    const float amax = __uint_as_float(gm);
    float inv = amax > 0.f ? 448.f / amax : 0.f;
    if (rfl(__builtin_amdgcn_raw_buffer_load_b32(self, TMO, 0, AUX_SYS)) == seq) inv = __uint_as_float(0x7fc00000u);
    if (lane == 0) {
      lds_st(smem, L_SCALE, __float_as_uint(inv));
      lds_st(smem, L_SFLAG, 1u);
    }
    STAMP(10);
  } else if (b < NPULLERS) {
    __builtin_amdgcn_s_setprio(3);
    const unsigned seq = rfl(__builtin_amdgcn_raw_buffer_load_b32(self, EPOCH, 0, AUX_SYS)) + 1;
    const unsigned pq = b % 7, k = b / 7;
    const unsigned p = (rank + 1 + pq) & 7;
    __amdgpu_buffer_rsrc_t rp = mk_rsrc(P.ws[p], P.ws_bytes);
    const unsigned rot = 2 * (pq + 1);
    const unsigned voff = lane * 16;
    auto job = [&](unsigned n, unsigned& src, unsigned& dst, unsigned& fsrc, unsigned& frdy) {
      const unsigned s = n / 12, r = n % 12;
      if (r < 4) {
        const unsigned rr = r * 16 + k, j = rr >> 2, qt = rr & 3;
        src = EXP_W + ((rank * 8 + s) * 16 + j) * 16384 + qt * 4096;
        dst = STG_W + ((p * 8 + s) * 16 + j) * 16384 + qt * 4096;
        fsrc = FW_SRC + (((p * 8 + s) * 16 + j) * 2 + (qt >> 1)) * 4;
        frdy = FW_RDY + (((p * 8 + s) * 16 + j) * 4 + qt) * 4;
      } else {
        const unsigned rr = (r - 4) * 16 + k, i = ((rr >> 3) + rot) & 15, sub = rr & 7;
        const unsigned idx = ((p * 8 + s) * 16 + i) * 8 + sub;
        src = dst = STG_X + idx * 4096;
        fsrc = FX_SRC + idx * 4;
        frdy = FX_RDY + idx * 4;
      }
    };
    STAMP(32);
    {
      unsigned src, dst, fsrc, frdy, nsrc, ndst, nfsrc, nfrdy;
      job(0, src, dst, fsrc, frdy);
      wait_flag(self, fsrc, seq);
      STAMP(33);
      u32x4 va[4], vb[4];
#pragma unroll
      for (int i = 0; i < 4; ++i) va[i] = __builtin_amdgcn_raw_buffer_load_b128(rp, voff + i * 1024, src, AUX_SYS);
      job(1, nsrc, ndst, nfsrc, nfrdy);
      unsigned fv = __builtin_amdgcn_raw_buffer_load_b32(self, nfsrc, 0, AUX_SYS);
      auto step = [&](u32x4 (&cur)[4], u32x4 (&nxt)[4], unsigned n) {
#pragma unroll
        for (int i = 0; i < 4; ++i) __builtin_amdgcn_raw_buffer_store_b128(cur[i], self, voff + i * 1024, dst, AUX_SC1);
        const unsigned myrdy = frdy;
        if (rfl(fv) < seq) wait_flag(self, nfsrc, seq);
#pragma unroll
        for (int i = 0; i < 4; ++i) nxt[i] = __builtin_amdgcn_raw_buffer_load_b128(rp, voff + i * 1024, nsrc, AUX_SYS);
        dst = ndst; frdy = nfrdy;
        job(n + 2 < 96 ? n + 2 : 95, nsrc, ndst, nfsrc, nfrdy);
        fv = __builtin_amdgcn_raw_buffer_load_b32(self, nfsrc, 0, AUX_SYS);
        wait_vm<5>();
        __builtin_amdgcn_raw_buffer_store_b32(seq, self, myrdy, 0, AUX_SC1);
        asm volatile("" ::: "memory");
      };
      for (unsigned n = 0; n < 94; n += 2) {
        step(va, vb, n);
        step(vb, va, n + 1);
      }
      step(va, vb, 94);
#pragma unroll
      for (int i = 0; i < 4; ++i) __builtin_amdgcn_raw_buffer_store_b128(vb[i], self, voff + i * 1024, dst, AUX_SC1);
      wait_vm<0>();
      __builtin_amdgcn_raw_buffer_store_b32(seq, self, frdy, 0, AUX_SC1);
    }
    STAMP(34);
    {
      __amdgpu_buffer_rsrc_t ro = mk_rsrc(P.out, (unsigned)MDIM * NDIM);
      const unsigned ovoff = (lane >> 3) * NDIM + (lane & 7) * 16;
      auto odst = [&](unsigned c) {
        const unsigned tb = c >> 3, w = c & 7, g = tb & 7, x = tb >> 3;
        const unsigned i = (g >> 1) * 4 + (x >> 3), j = (g & 1) * 8 + (x & 7);
        return (i * 256 + w * 32) * NDIM + p * NBLK + j * 128;
      };
      const u32x4 nanv = {0x7f7f7f7fu, 0x7f7f7f7fu, 0x7f7f7f7fu, 0x7f7f7f7fu};
      bool ok = wait_flag(self, FO_SRC + (p * 2048 + k) * 4, seq);
      STAMP(35);
      u32x4 v[4];
#pragma unroll
      for (int i = 0; i < 4; ++i) v[i] = __builtin_amdgcn_raw_buffer_load_b128(rp, voff + i * 1024, EXP_O + k * 4096, AUX_SYS);
      unsigned fv = __builtin_amdgcn_raw_buffer_load_b32(self, FO_SRC + (p * 2048 + k + 16) * 4, 0, AUX_SYS);
      for (unsigned n = 0; n < 128; ++n) {
        const unsigned c = k + 16 * n;
        const unsigned od = odst(c);
        if (!ok) {
#pragma unroll
          for (int i = 0; i < 4; ++i) v[i] = nanv;
        }
#pragma unroll
        for (int i = 0; i < 4; ++i) __builtin_amdgcn_raw_buffer_store_b128(v[i], ro, ovoff + i * 8 * NDIM, od, 0);
        if (n + 1 < 128) {
          if (rfl(fv) < seq) ok = wait_flag(self, FO_SRC + (p * 2048 + c + 16) * 4, seq) && ok;
#pragma unroll
          for (int i = 0; i < 4; ++i) v[i] = __builtin_amdgcn_raw_buffer_load_b128(rp, voff + i * 1024, EXP_O + (c + 16) * 4096, AUX_SYS);
          const unsigned c2 = n + 2 < 128 ? c + 32 : c + 16;
          fv = __builtin_amdgcn_raw_buffer_load_b32(self, FO_SRC + (p * 2048 + c2) * 4, 0, AUX_SYS);
        }
      }
    }
    STAMP(36);
  } else if (b == REDUCER_WG) {
    __builtin_amdgcn_s_setprio(3);
    const unsigned seq = rfl(__builtin_amdgcn_raw_buffer_load_b32(self, EPOCH, 0, AUX_SYS)) + 1;
    while (lds_ld(smem, L_QUEUE + 56 * 4) == 0) __builtin_amdgcn_s_sleep(8);
    unsigned gm = 0;
    unsigned long long t0 = now();
    for (unsigned spins = 0;; ++spins) {
      asm volatile("" ::: "memory");
      u32x4 a = __builtin_amdgcn_raw_buffer_load_b128(self, AMAX_WG + lane * 16, 0, AUX_SYS);
      u32x4 c = __builtin_amdgcn_raw_buffer_load_b128(self, AMAX_WG + 1024 + lane * 16, 0, AUX_SYS);
      gm = max(max(a[0], a[2]), max(c[0], c[2]));
      if (__all(a[1] >= seq && a[3] >= seq && c[1] >= seq && c[3] >= seq)) break;
      if ((spins & 63) == 63 && give_up(self, seq, t0)) break;
    }
#pragma unroll
    for (int o = 32; o >= 1; o >>= 1) gm = max(gm, (unsigned)__shfl_xor((int)gm, o));
    typedef __attribute__((address_space(1))) unsigned long long glb_u64;
    glb_u64* gp = (glb_u64*)(P.ws[lane >> 3] + AMAX_G + (lane & 7) * 128 + rank * 8);
    __hip_atomic_store(gp, ((unsigned long long)seq << 32) | gm, __ATOMIC_RELAXED, __HIP_MEMORY_SCOPE_SYSTEM);
    if (lane == 0) __builtin_amdgcn_raw_buffer_store_b32(seq, self, EPOCH, 0, AUX_SYS);
    STAMP(40);
  }
}

}

extern "C" void kernel_launch(int rank, int nranks, ncclComm_t comm, hipStream_t stream, void* const* d_in,
                              const int* in_sizes, int n_in, void* d_out, int out_size, const int* in_dtypes,
                              int out_dtype, void* const* d_out_all, void* const* d_ws_all, size_t ws_bytes) {
  (void)comm; (void)in_sizes; (void)in_dtypes; (void)out_dtype; (void)d_out_all; (void)out_size;
  if (nranks != NR || n_in < 2 || (unsigned)rank >= (unsigned)NR || ws_bytes < WS_NEED) {
    fprintf(stderr, "kernel_launch: unsupported configuration\n");
    return;
  }
  Params P;
  P.x = (const char*)d_in[0];
  P.w = (const char*)d_in[1];
  P.out = (char*)d_out;
  for (int p = 0; p < NR; ++p) P.ws[p] = (char*)d_ws_all[p];
  P.rank = rank;
  P.ws_bytes = WS_NEED;
  hipLaunchKernelGGL(fused_kernel, dim3(NWG), dim3(NTHREADS), 0, stream, P);
  hipError_t e = hipGetLastError();
  if (e != hipSuccess) fprintf(stderr, "launch failed: %s\n", hipGetErrorString(e));
}
```

```cpp
#include <hip/hip_runtime.h>
#include <rccl/rccl.h>
#include <cstdio>
#include <cstdint>
#include <type_traits>

namespace {

constexpr int NR = 8;
constexpr int MDIM = 4096, KSH = 512, NDIM = 16384, NBLK = 2048;
constexpr int NWG = 256;
constexpr int NTHREADS = 512;
#define NP_IN 16
#define NP_OUT 16
constexpr int NPULLERS = 7 * (NP_IN > NP_OUT ? NP_IN : NP_OUT);
constexpr int REDUCER = 127;
static_assert(NPULLERS <= REDUCER, "too many pullers");
#define PUSH_W 8
#define PUSH_O 16
#define POLL_SLEEP 4
#define EXPORT_DELAY 16
#define OUT_DELAY 8
constexpr unsigned long long DEADLINE_TICKS = 100000000ull;

constexpr unsigned STG_X = 0;
constexpr unsigned STG_W = 32u << 20;
constexpr unsigned EXP_W = 48u << 20;
constexpr unsigned EXP_O = 64u << 20;
constexpr unsigned FLG = 72u << 20;
constexpr unsigned FX_SRC = FLG;
constexpr unsigned FX_RDY = FLG + (64u << 10);
constexpr unsigned FW_SRC = FLG + (128u << 10);
constexpr unsigned FW_RDY = FLG + (144u << 10);
constexpr unsigned FO_SRC = FLG + (192u << 10);
constexpr unsigned AMAX_WG = FLG + (320u << 10);
constexpr unsigned AMAX_G = FLG + (324u << 10);
constexpr unsigned EPOCH = FLG + (328u << 10);
constexpr unsigned TMO = EPOCH + 256;
constexpr unsigned DUMMY = EPOCH + 512;
constexpr unsigned TICKET = FLG + (332u << 10);
constexpr unsigned DBG = FLG + (512u << 10);
constexpr unsigned WS_NEED = FLG + (1u << 20);

constexpr int SLOT_BYTES = 32 * 1024;
constexpr int L_CTRL = 4 * SLOT_BYTES;
constexpr int L_AMAX = L_CTRL;
constexpr int L_ACNT = L_CTRL + 4;
constexpr int L_SCALE = L_CTRL + 8;
constexpr int L_SFLAG = L_CTRL + 12;
constexpr int L_FLG = L_CTRL + 64;
constexpr int LDS_BYTES = L_FLG + 2048;

typedef __attribute__((ext_vector_type(8))) short bf16x8;
typedef __attribute__((ext_vector_type(4))) float f32x4;
typedef __attribute__((ext_vector_type(4))) unsigned u32x4;
typedef __attribute__((ext_vector_type(2))) unsigned u32x2;
typedef __attribute__((address_space(3))) unsigned lds_u32;
typedef __attribute__((address_space(1))) unsigned glb_u32;
typedef __attribute__((address_space(1))) unsigned long long glb_u64;

struct Params {
  const char* x;
  const char* w;
  char* out[NR];
  char* ws[NR];
  int rank;
};

constexpr int AUX_SC1 = 16, AUX_SYS = 17;
constexpr unsigned OUT_BYTES = (unsigned)MDIM * NDIM;

__device__ __forceinline__ __amdgpu_buffer_rsrc_t mk_rsrc(const void* p, unsigned bytes) {
  return __builtin_amdgcn_make_buffer_rsrc(const_cast<void*>(p), 0, bytes, 0x00020000);
}
template <int AUX> __device__ __forceinline__ void st128(u32x4 v, __amdgpu_buffer_rsrc_t r, unsigned off) {
  __builtin_amdgcn_raw_buffer_store_b128(v, r, off, 0, AUX);
}
template <int N> __device__ __forceinline__ void wait_vm() { asm volatile("s_waitcnt vmcnt(%0)" ::"i"(N) : "memory"); }
__device__ __forceinline__ void wait_lds() {
  asm volatile("" ::: "memory");
  __builtin_amdgcn_s_waitcnt(0xC07F);
  asm volatile("" ::: "memory");
}
__device__ __forceinline__ unsigned rfl(unsigned v) { return __builtin_amdgcn_readfirstlane(v); }
__device__ __forceinline__ unsigned lds_ld(const char* smem, int off) {
  return rfl(__hip_atomic_load((const lds_u32*)(smem + off), __ATOMIC_RELAXED, __HIP_MEMORY_SCOPE_WORKGROUP));
}
__device__ __forceinline__ void lds_st(char* smem, int off, unsigned v) {
  __hip_atomic_store((lds_u32*)(smem + off), v, __ATOMIC_RELAXED, __HIP_MEMORY_SCOPE_WORKGROUP);
}
__device__ __forceinline__ unsigned wave_max_u32(unsigned v) {
  v = max(v, (unsigned)__builtin_amdgcn_update_dpp(0, (int)v, 0x111, 0xf, 0xf, false));
  v = max(v, (unsigned)__builtin_amdgcn_update_dpp(0, (int)v, 0x112, 0xf, 0xf, false));
  v = max(v, (unsigned)__builtin_amdgcn_update_dpp(0, (int)v, 0x114, 0xf, 0xf, false));
  v = max(v, (unsigned)__builtin_amdgcn_update_dpp(0, (int)v, 0x118, 0xf, 0xf, false));
  const unsigned a = __builtin_amdgcn_readlane(v, 15), b = __builtin_amdgcn_readlane(v, 31);
  const unsigned c = __builtin_amdgcn_readlane(v, 47), d = __builtin_amdgcn_readlane(v, 63);
  return max(max(a, b), max(c, d));
}
__device__ __forceinline__ unsigned long long now() { return __builtin_amdgcn_s_memrealtime(); }
__device__ __forceinline__ unsigned long long pair(unsigned lo, unsigned hi) { return ((unsigned long long)hi << 32) | lo; }

#define STAMP(slot) do {} while (0)

__device__ __forceinline__ bool give_up(__amdgpu_buffer_rsrc_t self, unsigned seq, unsigned long long t0) {
  unsigned t = rfl(__builtin_amdgcn_raw_buffer_load_b32(self, TMO, 0, AUX_SYS));
  if (t == seq || now() - t0 > DEADLINE_TICKS) {
    __builtin_amdgcn_raw_buffer_store_b32(seq, self, TMO, 0, AUX_SYS);
    return true;
  }
  return false;
}

__device__ __forceinline__ bool wait_flag(__amdgpu_buffer_rsrc_t self, unsigned off, unsigned seq) {
  unsigned long long t0 = now();
  for (unsigned spins = 0;; ++spins) {
    asm volatile("" ::: "memory");
    u32x2 v = __builtin_amdgcn_raw_buffer_load_b64(self, off, 0, AUX_SYS);
    if (__all(v[0] >= seq && v[1] >= seq)) return true;
    if ((spins & 63) == 63 && give_up(self, seq, t0)) return false;
    __builtin_amdgcn_s_sleep(1);
  }
}

__device__ __forceinline__ u32x4 mk_desc(const void* p, unsigned bytes) {
  const unsigned long long a = (unsigned long long)p;
  return u32x4{(unsigned)a, (unsigned)(a >> 32) & 0xffffu, bytes, 0x00020000u};
}
template <int H>
__device__ __forceinline__ void glds22(unsigned voff, const char* wsrc, const char* xsrc, unsigned lw, unsigned lx) {
  unsigned keep;
  asm volatile(
      "s_mov_b32 %0, m0\n\ts_mov_b32 m0, %4\n\ts_nop 0\n\t"
      "global_load_lds_dwordx4 %1, %2 offset:%6 sc1\n\t"
      "global_load_lds_dwordx4 %1, %2 offset:%7 sc1\n\t"
      "s_mov_b32 m0, %5\n\ts_nop 0\n\t"
      "global_load_lds_dwordx4 %1, %3 offset:%6 sc1\n\t"
      "global_load_lds_dwordx4 %1, %3 offset:%7 sc1\n\t"
      "s_mov_b32 m0, %0"
      : "=&s"(keep) : "v"(voff), "s"(wsrc), "s"(xsrc), "s"(lw), "s"(lx), "i"(H * 2048), "i"(H * 2048 + 1024) : "memory");
}
__device__ __forceinline__ void a_ld4(u32x4 (&d)[4], unsigned voff, u32x4 rs, unsigned soff) {
  asm volatile(
      "buffer_load_dwordx4 %0, %4, %5, %6 offen sc0 sc1\n\t"
      "buffer_load_dwordx4 %1, %4, %5, %6 offen offset:1024 sc0 sc1\n\t"
      "buffer_load_dwordx4 %2, %4, %5, %6 offen offset:2048 sc0 sc1\n\t"
      "buffer_load_dwordx4 %3, %4, %5, %6 offen offset:3072 sc0 sc1"
      : "=&v"(d[0]), "=&v"(d[1]), "=&v"(d[2]), "=&v"(d[3]) : "v"(voff), "s"(rs), "s"(soff) : "memory");
}
__device__ __forceinline__ void a_st4_sc1(const u32x4 (&d)[4], unsigned voff, u32x4 rs, unsigned soff) {
  asm volatile(
      "buffer_store_dwordx4 %0, %4, %5, %6 offen sc1\n\t"
      "buffer_store_dwordx4 %1, %4, %5, %6 offen offset:1024 sc1\n\t"
      "buffer_store_dwordx4 %2, %4, %5, %6 offen offset:2048 sc1\n\t"
      "buffer_store_dwordx4 %3, %4, %5, %6 offen offset:3072 sc1\n\ts_nop 1"
      :: "v"(d[0]), "v"(d[1]), "v"(d[2]), "v"(d[3]), "v"(voff), "s"(rs), "s"(soff) : "memory");
}
__device__ __forceinline__ void a_st4_rows(const u32x4 (&d)[4], const unsigned (&vo)[4], u32x4 rs, unsigned soff) {
  asm volatile(
      "buffer_store_dwordx4 %0, %4, %8, %9 offen sc1\n\t"
      "buffer_store_dwordx4 %1, %5, %8, %9 offen sc1\n\t"
      "buffer_store_dwordx4 %2, %6, %8, %9 offen sc1\n\t"
      "buffer_store_dwordx4 %3, %7, %8, %9 offen sc1\n\ts_nop 1"
      :: "v"(d[0]), "v"(d[1]), "v"(d[2]), "v"(d[3]), "v"(vo[0]), "v"(vo[1]), "v"(vo[2]), "v"(vo[3]), "s"(rs), "s"(soff) : "memory");
}
__device__ __forceinline__ void a_ld2(u32x2& d, u32x4 rs, unsigned soff) {
  asm volatile("buffer_load_dwordx2 %0, off, %1, %2 sc0 sc1" : "=&v"(d) : "s"(rs), "s"(soff) : "memory");
}
__device__ __forceinline__ void glds_flag(unsigned voff, const char* base, unsigned l) {
  unsigned keep;
  asm volatile(
      "s_mov_b32 %0, m0\n\ts_mov_b32 m0, %3\n\ts_nop 0\n\t"
      "global_load_lds_dwordx4 %1, %2 sc0 sc1\n\t"
      "s_mov_b32 m0, %0"
      : "=&s"(keep) : "v"(voff), "s"(base), "s"(l) : "memory");
}
__device__ __forceinline__ void a_st2_sc1(u32x2 v, u32x4 rs, unsigned soff) {
  asm volatile("buffer_store_dwordx2 %0, off, %1, %2 sc1" :: "v"(v), "s"(rs), "s"(soff) : "memory");
}
__device__ __forceinline__ void a_ticket_flag(unsigned& tk, u32x2& fv, u32x4 rs, unsigned tk_off, unsigned f_off) {
  asm volatile(
      "s_mov_b64 exec, 1\n\t"
      "buffer_atomic_add %0, off, %2, %3 sc0\n\t"
      "s_mov_b64 exec, -1\n\t"
      "buffer_load_dwordx2 %1, off, %2, %4 sc0 sc1"
      : "+v"(tk), "+v"(fv) : "s"(rs), "s"(tk_off), "s"(f_off) : "memory");
}
template <int N> __device__ __forceinline__ void a_wait(u32x4 (&d)[4], unsigned& a, u32x2& b) {
  asm volatile("s_waitcnt vmcnt(%6)" : "+v"(d[0]), "+v"(d[1]), "+v"(d[2]), "+v"(d[3]), "+v"(a), "+v"(b) : "i"(N) : "memory");
}

__device__ __forceinline__ void export_job(const Params& P, unsigned n, unsigned lane, __amdgpu_buffer_rsrc_t self,
                                           unsigned& seq, bool first) {
  const unsigned rank = P.rank;
  const unsigned s = n / 384, r = n % 384;
  const unsigned sg = rank * 8 + s;
  if (r < 256) {
    const unsigned d = r & 7, j = r >> 4, h = (r >> 3) & 1;
    __amdgpu_buffer_rsrc_t rw = mk_rsrc(P.w, KSH * NDIM * 2);
    const unsigned o = lane & 15, g = lane >> 4;
    const unsigned voff = ((g * 8) * NDIM + o * 8) * 2;
    const unsigned soff = ((s * 64 + h * 32) * NDIM + d * NBLK + j * 128) * 2;
    u32x4 row[8];
#pragma unroll
    for (int jj = 0; jj < 8; ++jj) row[jj] = __builtin_amdgcn_raw_buffer_load_b128(rw, voff + jj * NDIM * 2, soff, 0);
    if (first) seq = rfl(__builtin_amdgcn_raw_buffer_load_b32(self, EPOCH, 0, AUX_SYS)) + 1;
    const bool direct = d == rank || (s == 0 && j < PUSH_W);
    char* const base = P.ws[direct ? d : rank];
    __amdgpu_buffer_rsrc_t rd = mk_rsrc(base, WS_NEED);
    const unsigned dst = (direct ? STG_W + sg * 16 * 16384 : EXP_W + (d * 8 + s) * 16 * 16384) + j * 16384 + h * 2048;
#pragma unroll
    for (int t = 0; t < 8; ++t) {
      u32x4 f;
#pragma unroll
      for (int e = 0; e < 4; ++e)
        f[e] = __builtin_amdgcn_perm(row[2 * e + 1][t >> 1], row[2 * e][t >> 1], (t & 1) ? 0x07060302u : 0x05040100u);
      st128<AUX_SYS>(f, rd, dst + lane * 16 + (t & 1) * 1024 + (t >> 1) * 4096);
    }
    wait_vm<0>();
    if (direct) {
      if (lane < 4) __builtin_amdgcn_raw_buffer_store_b32(seq, rd, FW_RDY + ((sg * 16 + j) * 4 + lane) * 8 + h * 4, 0, AUX_SYS);
    } else if (lane == 0) {
      __amdgpu_buffer_rsrc_t rp = mk_rsrc(P.ws[d], WS_NEED);
      __builtin_amdgcn_raw_buffer_store_b32(seq, rp, FW_SRC + (sg * 16 + j) * 8 + h * 4, 0, AUX_SYS);
    }
  } else {
    const unsigned i = (r - 256) >> 3, sub = (r - 256) & 7;
    __amdgpu_buffer_rsrc_t rx = mk_rsrc(P.x, MDIM * KSH * 2);
    const unsigned c = lane & 15, q = lane >> 4;
    const unsigned voff = (c * KSH + q * 8) * 2;
    const unsigned soff = ((i * 256 + sub * 32) * KSH + s * 64) * 2;
    u32x4 f[4];
#pragma unroll
    for (int b = 0; b < 4; ++b)
      f[b] = __builtin_amdgcn_raw_buffer_load_b128(rx, voff + ((b & 1) * 16 * KSH + (b >> 1) * 32) * 2, soff, 0);
    if (first) seq = rfl(__builtin_amdgcn_raw_buffer_load_b32(self, EPOCH, 0, AUX_SYS)) + 1;
    const unsigned idx = (sg * 16 + i) * 8 + sub;
#pragma unroll
    for (int b = 0; b < 4; ++b) st128<AUX_SYS>(f[b], self, STG_X + idx * 4096 + lane * 16 + b * 1024);
    wait_vm<0>();
    if (lane < NR) {
      glb_u64* fp = (glb_u64*)(P.ws[lane] + (lane == rank ? FX_RDY : FX_SRC) + idx * 8);
      __hip_atomic_store(fp, pair(seq, seq), __ATOMIC_RELAXED, __HIP_MEMORY_SCOPE_SYSTEM);
    }
  }
}

__global__ void __launch_bounds__(NTHREADS, 2) fused_kernel(Params P) {
  __shared__ __attribute__((aligned(16))) unsigned smem32[LDS_BYTES / 4];
  char* const smem = (char*)smem32;
  const unsigned lane = threadIdx.x & 63;
  const unsigned wave = rfl(threadIdx.x >> 6);
  const unsigned rank = P.rank;
  char* const wsl = P.ws[rank];
  __amdgpu_buffer_rsrc_t self = mk_rsrc(wsl, WS_NEED);
  const u32x4 ds = mk_desc(wsl, WS_NEED);
  const bool is_gemm = blockIdx.x >= 128;
  const unsigned b = blockIdx.x & 127;

  if (is_gemm) {
    const unsigned xg = b & 7, xi = b >> 3;
    const unsigned ti = (xg >> 1) * 4 + (xi >> 2), tj = (xg & 1) * 4 + (xi & 3);
    if (threadIdx.x < 16) smem32[L_CTRL / 4 + threadIdx.x] = 0;
    STAMP(wave == 0 ? 0 : 63);
    unsigned seq = 0;
    __builtin_amdgcn_s_sleep(EXPORT_DELAY);
    export_job(P, 384 + wave * 128 + b, lane, self, seq, true);
    export_job(P, 384 + 1024 + wave * 128 + b, lane, self, seq, false);
    if (wave < 5) export_job(P, 384 + 2048 + wave * 128 + b, lane, self, seq, false);
    if (wave == 0) STAMP(5);

    f32x4 acc[8][4];
#pragma unroll
    for (int t = 0; t < 8; ++t)
#pragma unroll
      for (int m = 0; m < 4; ++m) acc[t][m] = f32x4{0.f, 0.f, 0.f, 0.f};
    const unsigned lds0 = (unsigned)(uintptr_t)(lds_u32*)smem32 + wave * 2048;
    const unsigned voff = lane * 16;
    const char* xsrc = wsl + STG_X + (ti * 8 + wave) * 4096;
    const char* wsrc = wsl + STG_W + (tj * 8 + wave) * 4096;
    unsigned sp = 0;
    const unsigned fbase = (lane & 4) ? FW_RDY + ((lane >> 3) * 512 + tj * 8) * 8 + (lane & 3) * 16
                                      : FX_RDY + ((lane >> 3) * 1024 + ti * 8) * 8 + (lane & 3) * 16;
    const unsigned fsh = (lane & 4) ? 9 : 10;
    const unsigned lpad = (unsigned)(uintptr_t)(lds_u32*)smem32 + L_FLG;
    const unsigned lpadv = lpad + lane * 16;
    unsigned tick = 0;
    bool dead = false;
    auto request = [&]() {
      if (!dead) {
        glds_flag(fbase + ((sp & 7) << fsh), wsl, lpad + (tick & 1) * 1024);
      } else {
        const u32x4 all = {seq, seq, seq, seq};
        asm volatile("ds_write_b128 %0, %1\n\ts_waitcnt lgkmcnt(0)" :: "v"(lpadv + (tick & 1) * 1024), "v"(all) : "memory");
      }
    };
    unsigned issued = 0;
    const char* wcur = wsrc;
    const char* xcur = xsrc;
    const char* const padp = smem + L_FLG + lane * 16;
    auto snap = [&]() -> u32x4 { return *(const u32x4*)(padp + ((tick + 1) & 1) * 1024); };
    auto pick = [&](const u32x4 f) -> bool {
      unsigned long long m = __ballot(min(min(f[0], f[1]), min(f[2], f[3])) >= seq && sp < 8);
      m &= m >> 4; m &= m >> 2; m &= m >> 1; m &= 0x0101010101010101ull;
      if (m == 0) return false;
      const unsigned l8 = __builtin_ctzll(m);
      const unsigned sg = l8 + __builtin_amdgcn_readlane(sp, l8);
      sp += (lane >> 3) == (l8 >> 3) ? 1 : 0;
      wcur = wsrc + (size_t)sg * (64 * 4096);
      xcur = xsrc + (size_t)sg * (128 * 4096);
      return true;
    };
    auto issue_even = [&]() {
      const unsigned l = lds0 + (issued & 2) * SLOT_BYTES;
      glds22<0>(voff, wcur, xcur, l, l + 16384);
      ++issued;
    };
    auto issue_odd = [&]() {
      const unsigned l = lds0 + (issued & 2) * SLOT_BYTES + SLOT_BYTES - 2048;
      glds22<1>(voff, wcur, xcur, l, l + 16384);
      ++issued;
    };
    const char* wb = smem + (wave & 1) * 8192 + lane * 16;
    const char* xb = smem + 16384 + (wave >> 1) * 4096 + lane * 16;
    bf16x8 wa[8], xa[4], wc[8], xc[4];
    auto mmh = [&](bf16x8 (&wf)[8], bf16x8 (&xf)[4], auto h_tag) {
      constexpr int H = decltype(h_tag)::value;
      __builtin_amdgcn_sched_barrier(0);
#pragma unroll
      for (int t = 4 * H; t < 4 * H + 4; ++t)
#pragma unroll
        for (int m = 0; m < 4; ++m) acc[t][m] = __builtin_amdgcn_mfma_f32_16x16x32_bf16(wf[t], xf[m], acc[t][m], 0, 0, 0);
      __builtin_amdgcn_sched_barrier(0);
    };
    constexpr std::integral_constant<int, 0> H0{};
    constexpr std::integral_constant<int, 1> H1{};
    auto mm = [&](bf16x8 (&wf)[8], bf16x8 (&xf)[4]) { mmh(wf, xf, H0); mmh(wf, xf, H1); };
    auto rd = [&](bf16x8 (&wf)[8], bf16x8 (&xf)[4], unsigned so) {
#pragma unroll
      for (int m = 0; m < 4; ++m) xf[m] = *(const bf16x8*)(xb + so + m * 1024);
#pragma unroll
      for (int t = 0; t < 8; ++t) wf[t] = *(const bf16x8*)(wb + so + t * 1024);
    };
    unsigned hs = 0, young = 0;
    auto top = [&]() {
      const unsigned ahead = issued - hs;
      unsigned allow = ahead > 1 ? ahead - 1 : 0;
      if (wave == 0 && young < allow) allow = young;
      if (allow == 0) wait_vm<0>(); else if (allow == 1) wait_vm<4>(); else wait_vm<8>();
      __builtin_amdgcn_s_barrier();
      asm volatile("" ::: "memory");
    };
    auto stage = [&](const u32x4 f) {
      const unsigned lim = hs + 3 < 127 ? hs + 3 : 127, before = issued;
      const char* wodd = wcur;
      const char* xodd = xcur;
      bool got = false;
      if (issued + (issued & 1) <= lim && tick) got = pick(f);
      if (wave == 0) request();
      if ((issued & 1) && issued <= lim) {
        const unsigned l = lds0 + (issued & 2) * SLOT_BYTES + SLOT_BYTES - 2048;
        glds22<1>(voff, wodd, xodd, l, l + 16384);
        ++issued;
      }
      if (got) {
        issue_even();
        if (issued <= lim) issue_odd();
      }
      young = issued - before;
      ++tick;
    };
    auto rd_mm = [&](bf16x8 (&wf)[8], bf16x8 (&xf)[4], bf16x8 (&wp)[8], bf16x8 (&xp)[4]) {
      __builtin_amdgcn_sched_barrier(0);
      rd(wf, xf, (hs & 3) * SLOT_BYTES);
#pragma unroll
      for (int t = 0; t < 4; ++t)
#pragma unroll
        for (int m = 0; m < 4; ++m) acc[t][m] = __builtin_amdgcn_mfma_f32_16x16x32_bf16(wp[t], xp[m], acc[t][m], 0, 0, 0);
#pragma unroll
      for (int g = 0; g < 4; ++g) {
        __builtin_amdgcn_sched_group_barrier(0x008, 1, 0);
        __builtin_amdgcn_sched_group_barrier(0x100, 1, 0);
        __builtin_amdgcn_sched_group_barrier(0x008, 1, 0);
        __builtin_amdgcn_sched_group_barrier(0x100, 1, 0);
        __builtin_amdgcn_sched_group_barrier(0x008, 1, 0);
        __builtin_amdgcn_sched_group_barrier(0x100, 1, 0);
        __builtin_amdgcn_sched_group_barrier(0x008, 1, 0);
      }
      __builtin_amdgcn_sched_barrier(0);
    };
    bool deferred = false;
    while (hs < 128) {
      while (issued == hs + 3 && deferred && young == 1 && hs < 124) {
        if (wave == 0) wait_vm<4>(); else wait_vm<8>();
        __builtin_amdgcn_s_barrier();
        asm volatile("" ::: "memory");
        rd_mm(wa, xa, wc, xc);
        if (wave == 0) request();
        issue_odd();
        ++tick;
        mmh(wc, xc, H1);
        ++hs;
        wait_lds();
        if (wave == 0) wait_vm<4>(); else wait_vm<8>();
        __builtin_amdgcn_s_barrier();
        asm volatile("" ::: "memory");
        const u32x4 f = snap();
        rd_mm(wc, xc, wa, xa);
        const bool got = pick(f);
        if (wave == 0) request();
        if (got) issue_even();
        young = got ? 1 : 0;
        ++tick;
        mmh(wa, xa, H1);
        ++hs;
        wait_lds();
      }
      if (issued == hs) {
        if (deferred) mm(wc, xc);
        deferred = false;
        const unsigned long long t0 = now();
        for (unsigned idle = 1;; ++idle) {
          top();
          stage(snap());
          if (issued != hs) break;
          if (wave == 0 && (idle & 63) == 0 && give_up(self, seq, t0)) dead = true;
          __builtin_amdgcn_s_sleep(POLL_SLEEP);
        }
      }
      top();
      {
        const u32x4 f = snap();
        if (deferred) rd_mm(wa, xa, wc, xc); else rd(wa, xa, (hs & 3) * SLOT_BYTES);
        stage(f);
      }
      if (deferred) mmh(wc, xc, H1);
      wait_lds();
      __builtin_amdgcn_sched_barrier(0);
      ++hs;
      top();
      {
        const u32x4 f = snap();
        rd_mm(wc, xc, wa, xa);
        stage(f);
      }
      mmh(wa, xa, H1);
      wait_lds();
      __builtin_amdgcn_sched_barrier(0);
      ++hs;
      deferred = true;
    }
    mm(wc, xc);
    if (wave == 0) STAMP(2);
    float mx = 0.f;
#pragma unroll
    for (int t = 0; t < 8; ++t)
#pragma unroll
      for (int m = 0; m < 4; ++m)
#pragma unroll
        for (int r = 0; r < 4; ++r) mx = fmaxf(mx, fabsf(acc[t][m][r]));
    const unsigned wmx = wave_max_u32(__float_as_uint(mx));
    if (lane == 0) {
      __hip_atomic_fetch_max((lds_u32*)(smem + L_AMAX), wmx, __ATOMIC_RELAXED, __HIP_MEMORY_SCOPE_WORKGROUP);
      asm volatile("" ::: "memory");
      const unsigned cnt = __hip_atomic_fetch_add((lds_u32*)(smem + L_ACNT), 1u, __ATOMIC_RELAXED, __HIP_MEMORY_SCOPE_WORKGROUP);
      asm volatile("" ::: "memory");
      const unsigned am = __hip_atomic_load((lds_u32*)(smem + L_AMAX), __ATOMIC_RELAXED, __HIP_MEMORY_SCOPE_WORKGROUP);
      asm volatile("" ::: "memory");
      if (cnt == 7) {
        __builtin_amdgcn_raw_buffer_store_b64(u32x2{am, seq}, self, AMAX_WG + b * 8, 0, AUX_SYS);
        STAMP(9);
      }
    }
    if (wave == 0) {
      unsigned gm = 0, tm = 0;
      unsigned long long t0 = now();
      for (unsigned spins = 0;; ++spins) {
        asm volatile("" ::: "memory");
        u32x2 g = __builtin_amdgcn_raw_buffer_load_b64(self, AMAX_G + (b & 7) * 128 + (lane & 7) * 8, 0, AUX_SYS);
        tm = rfl(__builtin_amdgcn_raw_buffer_load_b32(self, TMO, 0, AUX_SYS));
        if (__all(g[1] >= seq)) { gm = g[0]; break; }
        if (tm == seq || ((spins & 63) == 63 && give_up(self, seq, t0))) { tm = seq; break; }
        __builtin_amdgcn_s_sleep(1);
      }
      const float amax = __uint_as_float(wave_max_u32(gm));
      float iv = amax > 0.f ? 448.f / amax : 0.f;
      if (tm == seq) iv = __uint_as_float(0x7fc00000u);
      if (lane == 0) {
        lds_st(smem, L_SCALE, __float_as_uint(iv));
        lds_st(smem, L_SFLAG, 1u);
      }
    }
    while (lds_ld(smem, L_SFLAG) == 0) __builtin_amdgcn_s_sleep(1);
    asm volatile("" ::: "memory");
    const float inv = __uint_as_float(lds_ld(smem, L_SCALE));
    if (wave == 0) STAMP(3);
    if (wave >= 1) __builtin_amdgcn_s_sleep(OUT_DELAY);
    char* tb = smem + wave * 8192;
    const unsigned c = lane & 15, q = lane >> 4;
    const unsigned ovoff = (lane >> 3) * NDIM + (lane & 7) * 16;
    const unsigned osoff = (ti * 256 + (wave >> 1) * 64) * NDIM + rank * NBLK + tj * 256 + (wave & 1) * 128;
    u32x4 ov[8];
#pragma unroll
    for (int hf = 0; hf < 2; ++hf) {
#pragma unroll
      for (int m = 2 * hf; m < 2 * hf + 2; ++m) {
        unsigned dw[8];
#pragma unroll
        for (int r = 0; r < 4; ++r)
#pragma unroll
          for (int hh = 0; hh < 2; ++hh) {
            int p = __builtin_amdgcn_cvt_pk_fp8_f32(acc[4 * hh + 0][m][r] * inv, acc[4 * hh + 1][m][r] * inv, 0, false);
            p = __builtin_amdgcn_cvt_pk_fp8_f32(acc[4 * hh + 2][m][r] * inv, acc[4 * hh + 3][m][r] * inv, p, true);
            dw[2 * r + hh] = (unsigned)p;
          }
        const unsigned row = m * 16 + c;
        *(u32x4*)(tb + row * 128 + (((2 * q) ^ (row & 7)) * 16)) = u32x4{dw[0], dw[1], dw[2], dw[3]};
        *(u32x4*)(tb + row * 128 + (((2 * q + 1) ^ (row & 7)) * 16)) = u32x4{dw[4], dw[5], dw[6], dw[7]};
      }
#pragma unroll
      for (int it = 4 * hf; it < 4 * hf + 4; ++it) {
        const unsigned row = it * 8 + (lane >> 3);
        ov[it] = *(const u32x4*)(tb + row * 128 + (((lane & 7) ^ (row & 7)) * 16));
      }
      const unsigned chunk = (wave * 2 + hf) * 128 + b;
      if (chunk < PUSH_O) {
        for (unsigned qq = 0; qq < 7; ++qq) {
          const unsigned p = (rank + 1 + (qq + b) % 7) & 7;
          __amdgpu_buffer_rsrc_t rp = mk_rsrc(P.out[p], OUT_BYTES);
#pragma unroll
          for (int it = 4 * hf; it < 4 * hf + 4; ++it) st128<AUX_SYS>(ov[it], rp, osoff + ovoff + it * 8 * NDIM);
        }
      } else {
#pragma unroll
        for (int it = 4 * hf; it < 4 * hf + 4; ++it)
          st128<AUX_SYS>(ov[it], self, EXP_O + chunk * 4096 + lane * 16 + (it & 3) * 1024);
      }
    }
    wait_vm<0>();
    if (lane < 2 * NR && (lane & 7) != rank) {
      glb_u64* fp = (glb_u64*)(P.ws[lane & 7] + FO_SRC + (rank * 2048 + (wave * 2 + (lane >> 3)) * 128 + b) * 8);
      __hip_atomic_store(fp, pair(seq, seq), __ATOMIC_RELAXED, __HIP_MEMORY_SCOPE_SYSTEM);
    }
    if (wave == 0) STAMP(4);
    __amdgpu_buffer_rsrc_t ro = mk_rsrc(P.out[rank], OUT_BYTES);
#pragma unroll
    for (int it = 0; it < 8; ++it) st128<AUX_SC1>(ov[it], ro, osoff + ovoff + it * 8 * NDIM);
  } else if (wave >= 1 && wave <= 3) {
    unsigned seq = 0;
    export_job(P, (wave - 1) * 128 + b, lane, self, seq, true);
    if (wave == 1) STAMP(1);
  } else if (wave == 0 && b < NPULLERS) {
    const unsigned seq = rfl(__builtin_amdgcn_raw_buffer_load_b32(self, EPOCH, 0, AUX_SYS)) + 1;
    const unsigned pq = b % 7, k = b / 7;
    const unsigned p = (rank + 1 + pq) & 7;
    const unsigned rot = 2 * (pq + 1);
    const unsigned voff = lane * 16;
    const u32x4 dp = mk_desc(P.ws[p], WS_NEED), dout = mk_desc(P.out[rank], OUT_BYTES);
    const unsigned ovoff = (lane >> 3) * NDIM + (lane & 7) * 16;
    const unsigned ovo[4] = {ovoff, ovoff + 8 * NDIM, ovoff + 16 * NDIM, ovoff + 24 * NDIM};
    const u32x2 seq2 = {seq, seq};
    const u32x4 nan4 = {0x7f7f7f7fu, 0x7f7f7f7fu, 0x7f7f7f7fu, 0x7f7f7f7fu};
    auto run = [&](auto phase_tag) {
      constexpr int PH = decltype(phase_tag)::value;
      constexpr unsigned J0 = PH == 0 ? 4 * PUSH_W : PUSH_O;
      constexpr unsigned NJ = PH == 0 ? 1536u : 2048u;
      constexpr unsigned NP = PH == 0 ? NP_IN : NP_OUT;
      if (k >= NP) return;
      const unsigned tk_off = TICKET + (PH * 8 + pq) * 128;
      const unsigned tk_base = (seq - 1) * (NJ - J0 - 2 * NP) - 3 * NP - J0;
      auto decode = [&](unsigned n, bool& is_w, unsigned& idx) {
        if (n < 64) { is_w = true; idx = p * 512 + n; return; }
        const unsigned s = (n - 64) / 192, r = (n - 64) % 192, t = r / 3, e = r % 3;
        is_w = s < 7 && e == 2;
        const unsigned xj = s < 7 ? t * 2 + e : r;
        idx = is_w ? (p * 8 + s + 1) * 64 + t : (p * 8 + s) * 128 + ((xj + rot * 8) & 127);
      };
      auto f_src = [&](unsigned n) -> unsigned {
        if constexpr (PH == 1) return FO_SRC + (p * 2048 + n) * 8;
        bool is_w; unsigned idx;
        decode(n, is_w, idx);
        return is_w ? FW_SRC + (idx >> 2) * 8 : FX_SRC + idx * 8;
      };
      auto addrs = [&](unsigned n, unsigned& src, unsigned& dst, unsigned& frdy) {
        if constexpr (PH == 1) {
          const unsigned tb = n & 127, w = n >> 8, g = tb & 7, x = tb >> 3;
          const unsigned i = (g >> 1) * 4 + (x >> 2), j = (g & 1) * 4 + (x & 3);
          src = EXP_O + n * 4096;
          dst = (i * 256 + (w >> 1) * 64 + ((n >> 7) & 1) * 32) * NDIM + p * NBLK + j * 256 + (w & 1) * 128;
          frdy = 0;
        } else {
          bool is_w; unsigned idx;
          decode(n, is_w, idx);
          dst = (is_w ? STG_W : STG_X) + idx * 4096;
          src = is_w ? EXP_W + (rank * 512 + (idx & 511)) * 4096 : dst;
          frdy = (is_w ? FW_RDY : FX_RDY) + idx * 8;
        }
      };
      u32x4 va[4], vb[4];
      unsigned j1 = J0 + k + NP, j2 = J0 + k + 2 * NP;
      unsigned src, dst, frdy, pfrdy = DUMMY;
      bool ok = wait_flag(self, f_src(J0 + k), seq);
      STAMP(PH == 0 ? 33 : 35);
      addrs(J0 + k, src, dst, frdy);
      u32x2 fv;
      unsigned dummy = 0;
      a_ld4(va, voff, dp, src);
      a_ld2(fv, ds, f_src(j1));
      unsigned nsrc, ndst, nfrdy;
      addrs(j1, nsrc, ndst, nfrdy);
      a_wait<0>(va, dummy, fv);
      auto step = [&](u32x4 (&cur)[4], u32x4 (&nxt)[4]) -> bool {
        if (j1 >= NJ) return false;
        const bool cur_ok = ok;
        if (rfl(min(fv[0], fv[1])) < seq) ok = wait_flag(self, f_src(j1), seq) && ok;
        a_ld4(nxt, voff, dp, nsrc);
        unsigned tk = 1;
        const bool has2 = j2 < NJ;
        if (has2) a_ticket_flag(tk, fv, ds, tk_off, f_src(j2));
        const unsigned cdst = dst, cpf = pfrdy;
        pfrdy = frdy; dst = ndst; frdy = nfrdy;
        if constexpr (PH == 0) {
          a_st4_sc1(cur, voff, ds, cdst);
        } else {
          if (!cur_ok) { cur[0] = nan4; cur[1] = nan4; cur[2] = nan4; cur[3] = nan4; }
          a_st4_rows(cur, ovo, dout, cdst);
        }
        j1 = j2;
        addrs(has2 ? j1 : J0, nsrc, ndst, nfrdy);
        asm volatile("" : "+s"(nsrc), "+s"(ndst), "+s"(nfrdy));
        if constexpr (PH == 0) {
          a_wait<0>(nxt, tk, fv);
          a_st2_sc1(seq2, ds, pfrdy);
        } else {
          a_wait<4>(nxt, tk, fv);
        }
        (void)cpf;
        j2 = has2 ? rfl(tk) - tk_base : NJ;
        if (j2 > NJ) j2 = NJ;
        return true;
      };
      auto last = [&](u32x4 (&cur)[4]) {
        if constexpr (PH == 0) {
          a_st4_sc1(cur, voff, ds, dst);
          wait_vm<0>();
          a_st2_sc1(seq2, ds, frdy);
        } else {
          if (!ok) { cur[0] = nan4; cur[1] = nan4; cur[2] = nan4; cur[3] = nan4; }
          a_st4_rows(cur, ovo, dout, dst);
        }
      };
      for (;;) {
        if (!step(va, vb)) { last(va); break; }
        if (!step(vb, va)) { last(vb); break; }
      }
    };
    STAMP(32);
    run(std::integral_constant<int, 0>{});
    STAMP(34);
    run(std::integral_constant<int, 1>{});
    if (PUSH_O > 0 && k == 0) {
      if (!wait_flag(self, FO_SRC + (p * 2048 + (lane % (PUSH_O > 0 ? PUSH_O : 1))) * 8, seq) && lane == 0)
        *(unsigned*)(P.out[rank] + p * NBLK) = 0x7f7f7f7fu;
    }
    STAMP(36);
  } else if (wave == 0 && b == REDUCER) {
    const unsigned seq = rfl(__builtin_amdgcn_raw_buffer_load_b32(self, EPOCH, 0, AUX_SYS)) + 1;
    unsigned gm = 0;
    unsigned long long t0 = now();
    for (unsigned spins = 0;; ++spins) {
      asm volatile("" ::: "memory");
      u32x4 a = __builtin_amdgcn_raw_buffer_load_b128(self, AMAX_WG + lane * 16, 0, AUX_SYS);
      gm = max(a[0], a[2]);
      if (__all(a[1] >= seq && a[3] >= seq)) break;
      if ((spins & 63) == 63 && give_up(self, seq, t0)) break;
    }
    gm = wave_max_u32(gm);
    glb_u64* gp = (glb_u64*)(P.ws[lane >> 3] + AMAX_G + (lane & 7) * 128 + rank * 8);
    __hip_atomic_store(gp, pair(gm, seq), __ATOMIC_RELAXED, __HIP_MEMORY_SCOPE_SYSTEM);
    if (lane == 0) __builtin_amdgcn_raw_buffer_store_b32(seq, self, EPOCH, 0, AUX_SYS);
    STAMP(39);
  }
}

}

extern "C" void kernel_launch(int rank, int nranks, ncclComm_t comm, hipStream_t stream, void* const* d_in,
                              const int* in_sizes, int n_in, void* d_out, int out_size, const int* in_dtypes,
                              int out_dtype, void* const* d_out_all, void* const* d_ws_all, size_t ws_bytes) {
  (void)comm; (void)in_sizes; (void)in_dtypes; (void)out_dtype; (void)out_size;
  if (nranks != NR || n_in < 2 || (unsigned)rank >= (unsigned)NR || ws_bytes < WS_NEED) {
    fprintf(stderr, "kernel_launch: unsupported configuration\n");
    return;
  }
  Params P;
  P.x = (const char*)d_in[0];
  P.w = (const char*)d_in[1];
  for (int p = 0; p < NR; ++p) {
    P.out[p] = (char*)d_out_all[p];
    P.ws[p] = (char*)d_ws_all[p];
  }
  P.out[rank] = (char*)d_out;
  P.rank = rank;
  hipLaunchKernelGGL(fused_kernel, dim3(NWG), dim3(NTHREADS), 0, stream, P);
  hipError_t e = hipGetLastError();
  if (e != hipSuccess) fprintf(stderr, "launch failed: %s\n", hipGetErrorString(e));
}
```

```cpp
#include <hip/hip_runtime.h>
#include <rccl/rccl.h>
#include <cstdio>
#include <cstdint>
#include <type_traits>

namespace {

constexpr int NR = 8;
constexpr int MDIM = 4096, KSH = 512, NDIM = 16384, NBLK = 2048;
constexpr int NWG = 256;
constexpr int NTHREADS = 512;
#define NP_IN 16
#define NP_OUT 16
constexpr int NPULLERS = 7 * (NP_IN > NP_OUT ? NP_IN : NP_OUT);
constexpr int REDUCER = 127;
static_assert(NPULLERS <= REDUCER, "too many pullers");
#define PUSH_W 12
#define PUSH_O 16
#define PACE_W 3
#define YROWS 160
#define POLL_SLEEP 4
#define EXPORT_DELAY 16
#define OUT_DELAY 8
constexpr unsigned long long DEADLINE_TICKS = 100000000ull;

constexpr unsigned STG_X = 0;
constexpr unsigned STG_W = 32u << 20;
constexpr unsigned EXP_W = 48u << 20;
constexpr unsigned EXP_O = 64u << 20;
constexpr unsigned FLG = 72u << 20;
constexpr unsigned FX_SRC = FLG;
constexpr unsigned FX_RDY = FLG + (64u << 10);
constexpr unsigned FW_SRC = FLG + (128u << 10);
constexpr unsigned FW_RDY = FLG + (144u << 10);
constexpr unsigned FO_SRC = FLG + (192u << 10);
constexpr unsigned AMAX_WG = FLG + (320u << 10);
constexpr unsigned AMAX_G = FLG + (324u << 10);
constexpr unsigned EPOCH = FLG + (328u << 10);
constexpr unsigned TMO = EPOCH + 256;
constexpr unsigned DUMMY = EPOCH + 512;
constexpr unsigned TICKET = FLG + (332u << 10);
constexpr unsigned FY_SRC = FLG + (336u << 10);
constexpr unsigned FY_RDY = FLG + (352u << 10);
constexpr unsigned DBG = FLG + (512u << 10);
constexpr unsigned EXP_Y = FLG + (1u << 20);
constexpr unsigned STG_Y = FLG + (2u << 20);
constexpr unsigned WS_NEED = FLG + (10u << 20);

constexpr int SLOT_BYTES = 32 * 1024;
constexpr int L_CTRL = 4 * SLOT_BYTES;
constexpr int L_AMAX = L_CTRL;
constexpr int L_ACNT = L_CTRL + 4;
constexpr int L_SCALE = L_CTRL + 8;
constexpr int L_SFLAG = L_CTRL + 12;
constexpr int L_FLG = L_CTRL + 64;
constexpr int LDS_BYTES = L_FLG + 2048;

typedef __attribute__((ext_vector_type(8))) short bf16x8;
typedef __attribute__((ext_vector_type(4))) float f32x4;
typedef __attribute__((ext_vector_type(4))) unsigned u32x4;
typedef __attribute__((ext_vector_type(2))) unsigned u32x2;
typedef __attribute__((ext_vector_type(2))) float f32x2;
typedef __attribute__((ext_vector_type(2))) __bf16 bf16x2;
typedef __attribute__((address_space(3))) unsigned lds_u32;
typedef __attribute__((address_space(1))) unsigned glb_u32;
typedef __attribute__((address_space(1))) unsigned long long glb_u64;

struct Params {
  const char* x;
  const char* w;
  char* out[NR];
  char* ws[NR];
  int rank;
};

constexpr int AUX_SC1 = 16, AUX_SYS = 17;
constexpr unsigned OUT_BYTES = (unsigned)MDIM * NDIM;

__device__ __forceinline__ __amdgpu_buffer_rsrc_t mk_rsrc(const void* p, unsigned bytes) {
  return __builtin_amdgcn_make_buffer_rsrc(const_cast<void*>(p), 0, bytes, 0x00020000);
}
template <int AUX> __device__ __forceinline__ void st128(u32x4 v, __amdgpu_buffer_rsrc_t r, unsigned off) {
  __builtin_amdgcn_raw_buffer_store_b128(v, r, off, 0, AUX);
}
template <int N> __device__ __forceinline__ void wait_vm() { asm volatile("s_waitcnt vmcnt(%0)" ::"i"(N) : "memory"); }
__device__ __forceinline__ void wait_lds() {
  asm volatile("" ::: "memory");
  __builtin_amdgcn_s_waitcnt(0xC07F);
  asm volatile("" ::: "memory");
}
__device__ __forceinline__ unsigned rfl(unsigned v) { return __builtin_amdgcn_readfirstlane(v); }
__device__ __forceinline__ unsigned lds_ld(const char* smem, int off) {
  return rfl(__hip_atomic_load((const lds_u32*)(smem + off), __ATOMIC_RELAXED, __HIP_MEMORY_SCOPE_WORKGROUP));
}
__device__ __forceinline__ void lds_st(char* smem, int off, unsigned v) {
  __hip_atomic_store((lds_u32*)(smem + off), v, __ATOMIC_RELAXED, __HIP_MEMORY_SCOPE_WORKGROUP);
}
__device__ __forceinline__ unsigned wave_max_u32(unsigned v) {
  v = max(v, (unsigned)__builtin_amdgcn_update_dpp(0, (int)v, 0x111, 0xf, 0xf, false));
  v = max(v, (unsigned)__builtin_amdgcn_update_dpp(0, (int)v, 0x112, 0xf, 0xf, false));
  v = max(v, (unsigned)__builtin_amdgcn_update_dpp(0, (int)v, 0x114, 0xf, 0xf, false));
  v = max(v, (unsigned)__builtin_amdgcn_update_dpp(0, (int)v, 0x118, 0xf, 0xf, false));
  const unsigned a = __builtin_amdgcn_readlane(v, 15), b = __builtin_amdgcn_readlane(v, 31);
  const unsigned c = __builtin_amdgcn_readlane(v, 47), d = __builtin_amdgcn_readlane(v, 63);
  return max(max(a, b), max(c, d));
}
__device__ __forceinline__ unsigned long long now() { return __builtin_amdgcn_s_memrealtime(); }
__device__ __forceinline__ unsigned long long pair(unsigned lo, unsigned hi) { return ((unsigned long long)hi << 32) | lo; }

#define STAMP(slot) do {} while (0)

__device__ __forceinline__ bool give_up(__amdgpu_buffer_rsrc_t self, unsigned seq, unsigned long long t0) {
  unsigned t = rfl(__builtin_amdgcn_raw_buffer_load_b32(self, TMO, 0, AUX_SYS));
  if (t == seq || now() - t0 > DEADLINE_TICKS) {
    __builtin_amdgcn_raw_buffer_store_b32(seq, self, TMO, 0, AUX_SYS);
    return true;
  }
  return false;
}

__device__ __forceinline__ bool wait_flag(__amdgpu_buffer_rsrc_t self, unsigned off, unsigned seq) {
  unsigned long long t0 = now();
  for (unsigned spins = 0;; ++spins) {
    asm volatile("" ::: "memory");
    u32x2 v = __builtin_amdgcn_raw_buffer_load_b64(self, off, 0, AUX_SYS);
    if (__all(v[0] >= seq && v[1] >= seq)) return true;
    if ((spins & 63) == 63 && give_up(self, seq, t0)) return false;
    __builtin_amdgcn_s_sleep(1);
  }
}

__device__ __forceinline__ u32x4 mk_desc(const void* p, unsigned bytes) {
  const unsigned long long a = (unsigned long long)p;
  return u32x4{(unsigned)a, (unsigned)(a >> 32) & 0xffffu, bytes, 0x00020000u};
}
template <int H>
__device__ __forceinline__ void glds22(unsigned voff, const char* wsrc, const char* xsrc, unsigned lw, unsigned lx) {
  unsigned keep;
  asm volatile(
      "s_mov_b32 %0, m0\n\ts_mov_b32 m0, %4\n\ts_nop 0\n\t"
      "global_load_lds_dwordx4 %1, %2 offset:%6 sc1\n\t"
      "global_load_lds_dwordx4 %1, %2 offset:%7 sc1\n\t"
      "s_mov_b32 m0, %5\n\ts_nop 0\n\t"
      "global_load_lds_dwordx4 %1, %3 offset:%6 sc1\n\t"
      "global_load_lds_dwordx4 %1, %3 offset:%7 sc1\n\t"
      "s_mov_b32 m0, %0"
      : "=&s"(keep) : "v"(voff), "s"(wsrc), "s"(xsrc), "s"(lw), "s"(lx), "i"(H * 2048), "i"(H * 2048 + 1024) : "memory");
}
__device__ __forceinline__ void a_ld4(u32x4 (&d)[4], unsigned voff, u32x4 rs, unsigned soff) {
  asm volatile(
      "buffer_load_dwordx4 %0, %4, %5, %6 offen sc0 sc1\n\t"
      "buffer_load_dwordx4 %1, %4, %5, %6 offen offset:1024 sc0 sc1\n\t"
      "buffer_load_dwordx4 %2, %4, %5, %6 offen offset:2048 sc0 sc1\n\t"
      "buffer_load_dwordx4 %3, %4, %5, %6 offen offset:3072 sc0 sc1"
      : "=&v"(d[0]), "=&v"(d[1]), "=&v"(d[2]), "=&v"(d[3]) : "v"(voff), "s"(rs), "s"(soff) : "memory");
}
__device__ __forceinline__ void a_st4_sc1(const u32x4 (&d)[4], unsigned voff, u32x4 rs, unsigned soff) {
  asm volatile(
      "buffer_store_dwordx4 %0, %4, %5, %6 offen sc1\n\t"
      "buffer_store_dwordx4 %1, %4, %5, %6 offen offset:1024 sc1\n\t"
      "buffer_store_dwordx4 %2, %4, %5, %6 offen offset:2048 sc1\n\t"
      "buffer_store_dwordx4 %3, %4, %5, %6 offen offset:3072 sc1\n\ts_nop 1"
      :: "v"(d[0]), "v"(d[1]), "v"(d[2]), "v"(d[3]), "v"(voff), "s"(rs), "s"(soff) : "memory");
}
__device__ __forceinline__ void a_st4_rows(const u32x4 (&d)[4], const unsigned (&vo)[4], u32x4 rs, unsigned soff) {
  asm volatile(
      "buffer_store_dwordx4 %0, %4, %8, %9 offen sc1\n\t"
      "buffer_store_dwordx4 %1, %5, %8, %9 offen sc1\n\t"
      "buffer_store_dwordx4 %2, %6, %8, %9 offen sc1\n\t"
      "buffer_store_dwordx4 %3, %7, %8, %9 offen sc1\n\ts_nop 1"
      :: "v"(d[0]), "v"(d[1]), "v"(d[2]), "v"(d[3]), "v"(vo[0]), "v"(vo[1]), "v"(vo[2]), "v"(vo[3]), "s"(rs), "s"(soff) : "memory");
}
__device__ __forceinline__ void a_ld2(u32x2& d, u32x4 rs, unsigned soff) {
  asm volatile("buffer_load_dwordx2 %0, off, %1, %2 sc0 sc1" : "=&v"(d) : "s"(rs), "s"(soff) : "memory");
}
__device__ __forceinline__ void glds_flag(unsigned voff, const char* base, unsigned l) {
  unsigned keep;
  asm volatile(
      "s_mov_b32 %0, m0\n\ts_mov_b32 m0, %3\n\ts_nop 0\n\t"
      "global_load_lds_dwordx4 %1, %2 sc0 sc1\n\t"
      "s_mov_b32 m0, %0"
      : "=&s"(keep) : "v"(voff), "s"(base), "s"(l) : "memory");
}
__device__ __forceinline__ void a_st2_sc1(u32x2 v, u32x4 rs, unsigned soff) {
  asm volatile("buffer_store_dwordx2 %0, off, %1, %2 sc1" :: "v"(v), "s"(rs), "s"(soff) : "memory");
}
__device__ __forceinline__ void a_ticket_flag(unsigned& tk, u32x2& fv, u32x4 rs, unsigned tk_off, unsigned f_off) {
  asm volatile(
      "s_mov_b64 exec, 1\n\t"
      "buffer_atomic_add %0, off, %2, %3 sc0\n\t"
      "s_mov_b64 exec, -1\n\t"
      "buffer_load_dwordx2 %1, off, %2, %4 sc0 sc1"
      : "+v"(tk), "+v"(fv) : "s"(rs), "s"(tk_off), "s"(f_off) : "memory");
}
template <int N> __device__ __forceinline__ void a_wait(u32x4 (&d)[4], unsigned& a, u32x2& b) {
  asm volatile("s_waitcnt vmcnt(%6)" : "+v"(d[0]), "+v"(d[1]), "+v"(d[2]), "+v"(d[3]), "+v"(a), "+v"(b) : "i"(N) : "memory");
}

__device__ __forceinline__ void export_job(const Params& P, unsigned n, unsigned lane, __amdgpu_buffer_rsrc_t self,
                                           unsigned& seq, bool first) {
  const unsigned rank = P.rank;
  const unsigned s = n / 384, r = n % 384;
  const unsigned sg = rank * 8 + s;
  if (r < 256) {
    const unsigned d = r & 7, j = r >> 4, h = (r >> 3) & 1;
    __amdgpu_buffer_rsrc_t rw = mk_rsrc(P.w, KSH * NDIM * 2);
    const unsigned o = lane & 15, g = lane >> 4;
    const unsigned voff = ((g * 8) * NDIM + o * 8) * 2;
    const unsigned soff = ((s * 64 + h * 32) * NDIM + d * NBLK + j * 128) * 2;
    u32x4 row[8];
#pragma unroll
    for (int jj = 0; jj < 8; ++jj) row[jj] = __builtin_amdgcn_raw_buffer_load_b128(rw, voff + jj * NDIM * 2, soff, 0);
    if (first) seq = rfl(__builtin_amdgcn_raw_buffer_load_b32(self, EPOCH, 0, AUX_SYS)) + 1;
    const bool direct = d == rank || (s == 0 && j < PUSH_W);
    char* const base = P.ws[direct ? d : rank];
    __amdgpu_buffer_rsrc_t rd = mk_rsrc(base, WS_NEED);
    const unsigned dst = (direct ? STG_W + sg * 16 * 16384 : EXP_W + (d * 8 + s) * 16 * 16384) + j * 16384 + h * 2048;
#pragma unroll
    for (int t = 0; t < 8; ++t) {
      u32x4 f;
#pragma unroll
      for (int e = 0; e < 4; ++e)
        f[e] = __builtin_amdgcn_perm(row[2 * e + 1][t >> 1], row[2 * e][t >> 1], (t & 1) ? 0x07060302u : 0x05040100u);
      st128<AUX_SYS>(f, rd, dst + lane * 16 + (t & 1) * 1024 + (t >> 1) * 4096);
      if (direct && d != rank) wait_vm<PACE_W - 1>();
    }
    wait_vm<0>();
    if (direct) {
      if (lane < 4) __builtin_amdgcn_raw_buffer_store_b32(seq, rd, FW_RDY + ((sg * 16 + j) * 4 + lane) * 8 + h * 4, 0, AUX_SYS);
    } else if (lane == 0) {
      __amdgpu_buffer_rsrc_t rp = mk_rsrc(P.ws[d], WS_NEED);
      __builtin_amdgcn_raw_buffer_store_b32(seq, rp, FW_SRC + (sg * 16 + j) * 8 + h * 4, 0, AUX_SYS);
    }
  } else {
    const unsigned i = (r - 256) >> 3, sub = (r - 256) & 7;
    __amdgpu_buffer_rsrc_t rx = mk_rsrc(P.x, MDIM * KSH * 2);
    const unsigned c = lane & 15, q = lane >> 4;
    const unsigned voff = (c * KSH + q * 8) * 2;
    const unsigned soff = ((i * 256 + sub * 32) * KSH + s * 64) * 2;
    u32x4 f[4];
#pragma unroll
    for (int b = 0; b < 4; ++b)
      f[b] = __builtin_amdgcn_raw_buffer_load_b128(rx, voff + ((b & 1) * 16 * KSH + (b >> 1) * 32) * 2, soff, 0);
    if (first) seq = rfl(__builtin_amdgcn_raw_buffer_load_b32(self, EPOCH, 0, AUX_SYS)) + 1;
    const unsigned idx = (sg * 16 + i) * 8 + sub;
#pragma unroll
    for (int b = 0; b < 4; ++b) st128<AUX_SYS>(f[b], self, STG_X + idx * 4096 + lane * 16 + b * 1024);
    wait_vm<0>();
    if (lane < NR) {
      glb_u64* fp = (glb_u64*)(P.ws[lane] + (lane == rank ? FX_RDY : FX_SRC) + idx * 8);
      __hip_atomic_store(fp, pair(seq, seq), __ATOMIC_RELAXED, __HIP_MEMORY_SCOPE_SYSTEM);
    }
  }
}

__global__ void __launch_bounds__(NTHREADS, 2) fused_kernel(Params P) {
  __shared__ __attribute__((aligned(16))) unsigned smem32[LDS_BYTES / 4];
  char* const smem = (char*)smem32;
  const unsigned lane = threadIdx.x & 63;
  const unsigned wave = rfl(threadIdx.x >> 6);
  const unsigned rank = P.rank;
  char* const wsl = P.ws[rank];
  __amdgpu_buffer_rsrc_t self = mk_rsrc(wsl, WS_NEED);
  const u32x4 ds = mk_desc(wsl, WS_NEED);
  const bool is_gemm = blockIdx.x >= 128;
  const unsigned b = blockIdx.x & 127;

  if (is_gemm) {
    const unsigned xg = b & 7, xi = b >> 3;
    const unsigned ti = (xg >> 1) * 4 + (xi >> 2), tj = (xg & 1) * 4 + (xi & 3);
    if (threadIdx.x < 16) smem32[L_CTRL / 4 + threadIdx.x] = 0;
    STAMP(wave == 0 ? 0 : 63);
    unsigned seq = 0;
    __builtin_amdgcn_s_sleep(EXPORT_DELAY);
    export_job(P, 384 + wave * 128 + b, lane, self, seq, true);
    export_job(P, 384 + 1024 + wave * 128 + b, lane, self, seq, false);
    if (wave < 5) export_job(P, 384 + 2048 + wave * 128 + b, lane, self, seq, false);
    if (wave == 0) STAMP(5);

    f32x4 acc[8][4];
#pragma unroll
    for (int t = 0; t < 8; ++t)
#pragma unroll
      for (int m = 0; m < 4; ++m) acc[t][m] = f32x4{0.f, 0.f, 0.f, 0.f};
    const unsigned lds0 = (unsigned)(uintptr_t)(lds_u32*)smem32 + wave * 2048;
    const unsigned voff = lane * 16;
    const char* xsrc = wsl + STG_X + (ti * 8 + wave) * 4096;
    const char* wsrc = wsl + STG_W + (tj * 8 + wave) * 4096;
    unsigned sp = 0;
    const unsigned fbase = (lane & 4) ? FW_RDY + ((lane >> 3) * 512 + tj * 8) * 8 + (lane & 3) * 16
                                      : FX_RDY + ((lane >> 3) * 1024 + ti * 8) * 8 + (lane & 3) * 16;
    const unsigned fsh = (lane & 4) ? 9 : 10;
    const unsigned lpad = (unsigned)(uintptr_t)(lds_u32*)smem32 + L_FLG;
    const unsigned lpadv = lpad + lane * 16;
    unsigned tick = 0;
    bool dead = false;
    auto request = [&]() {
      if (!dead) {
        glds_flag(fbase + ((sp & 7) << fsh), wsl, lpad + (tick & 1) * 1024);
      } else {
        const u32x4 all = {seq, seq, seq, seq};
        asm volatile("ds_write_b128 %0, %1\n\ts_waitcnt lgkmcnt(0)" :: "v"(lpadv + (tick & 1) * 1024), "v"(all) : "memory");
      }
    };
    unsigned issued = 0;
    const char* wcur = wsrc;
    const char* xcur = xsrc;
    const char* const padp = smem + L_FLG + lane * 16;
    auto snap = [&]() -> u32x4 { return *(const u32x4*)(padp + ((tick + 1) & 1) * 1024); };
    auto pick = [&](const u32x4 f) -> bool {
      unsigned long long m = __ballot(min(min(f[0], f[1]), min(f[2], f[3])) >= seq && sp < 8);
      m &= m >> 4; m &= m >> 2; m &= m >> 1; m &= 0x0101010101010101ull;
      if (m == 0) return false;
      const unsigned l8 = __builtin_ctzll(m);
      const unsigned sg = l8 + __builtin_amdgcn_readlane(sp, l8);
      sp += (lane >> 3) == (l8 >> 3) ? 1 : 0;
      wcur = wsrc + (size_t)sg * (64 * 4096);
      xcur = xsrc + (size_t)sg * (128 * 4096);
      return true;
    };
    auto issue_even = [&]() {
      const unsigned l = lds0 + (issued & 2) * SLOT_BYTES;
      glds22<0>(voff, wcur, xcur, l, l + 16384);
      ++issued;
    };
    auto issue_odd = [&]() {
      const unsigned l = lds0 + (issued & 2) * SLOT_BYTES + SLOT_BYTES - 2048;
      glds22<1>(voff, wcur, xcur, l, l + 16384);
      ++issued;
    };
    const char* wb = smem + (wave & 1) * 8192 + lane * 16;
    const char* xb = smem + 16384 + (wave >> 1) * 4096 + lane * 16;
    bf16x8 wa[8], xa[4], wc[8], xc[4];
    auto mmh = [&](bf16x8 (&wf)[8], bf16x8 (&xf)[4], auto h_tag) {
      constexpr int H = decltype(h_tag)::value;
      __builtin_amdgcn_sched_barrier(0);
#pragma unroll
      for (int t = 4 * H; t < 4 * H + 4; ++t)
#pragma unroll
        for (int m = 0; m < 4; ++m) acc[t][m] = __builtin_amdgcn_mfma_f32_16x16x32_bf16(wf[t], xf[m], acc[t][m], 0, 0, 0);
      __builtin_amdgcn_sched_barrier(0);
    };
    constexpr std::integral_constant<int, 0> H0{};
    constexpr std::integral_constant<int, 1> H1{};
    auto mm = [&](bf16x8 (&wf)[8], bf16x8 (&xf)[4]) { mmh(wf, xf, H0); mmh(wf, xf, H1); };
    auto rd = [&](bf16x8 (&wf)[8], bf16x8 (&xf)[4], unsigned so) {
#pragma unroll
      for (int m = 0; m < 4; ++m) xf[m] = *(const bf16x8*)(xb + so + m * 1024);
#pragma unroll
      for (int t = 0; t < 8; ++t) wf[t] = *(const bf16x8*)(wb + so + t * 1024);
    };
    unsigned hs = 0, young = 0;
    auto top = [&]() {
      const unsigned ahead = issued - hs;
      unsigned allow = ahead > 1 ? ahead - 1 : 0;
      if (wave == 0 && young < allow) allow = young;
      if (allow == 0) wait_vm<0>(); else if (allow == 1) wait_vm<4>(); else wait_vm<8>();
      __builtin_amdgcn_s_barrier();
      asm volatile("" ::: "memory");
    };
    auto stage = [&](const u32x4 f) {
      const unsigned lim = hs + 3 < 127 ? hs + 3 : 127, before = issued;
      const char* wodd = wcur;
      const char* xodd = xcur;
      bool got = false;
      if (issued + (issued & 1) <= lim && tick) got = pick(f);
      if (wave == 0) request();
      if ((issued & 1) && issued <= lim) {
        const unsigned l = lds0 + (issued & 2) * SLOT_BYTES + SLOT_BYTES - 2048;
        glds22<1>(voff, wodd, xodd, l, l + 16384);
        ++issued;
      }
      if (got) {
        issue_even();
        if (issued <= lim) issue_odd();
      }
      young = issued - before;
      ++tick;
    };
    auto rd_mm = [&](bf16x8 (&wf)[8], bf16x8 (&xf)[4], bf16x8 (&wp)[8], bf16x8 (&xp)[4]) {
      __builtin_amdgcn_sched_barrier(0);
      rd(wf, xf, (hs & 3) * SLOT_BYTES);
#pragma unroll
      for (int t = 0; t < 4; ++t)
#pragma unroll
        for (int m = 0; m < 4; ++m) acc[t][m] = __builtin_amdgcn_mfma_f32_16x16x32_bf16(wp[t], xp[m], acc[t][m], 0, 0, 0);
#pragma unroll
      for (int g = 0; g < 4; ++g) {
        __builtin_amdgcn_sched_group_barrier(0x008, 1, 0);
        __builtin_amdgcn_sched_group_barrier(0x100, 1, 0);
        __builtin_amdgcn_sched_group_barrier(0x008, 1, 0);
        __builtin_amdgcn_sched_group_barrier(0x100, 1, 0);
        __builtin_amdgcn_sched_group_barrier(0x008, 1, 0);
        __builtin_amdgcn_sched_group_barrier(0x100, 1, 0);
        __builtin_amdgcn_sched_group_barrier(0x008, 1, 0);
      }
      __builtin_amdgcn_sched_barrier(0);
    };
    bool deferred = false;
    while (hs < 128) {
      while (issued == hs + 3 && deferred && young == 1 && hs < 124) {
        wait_vm<8>();
        __builtin_amdgcn_s_barrier();
        asm volatile("" ::: "memory");
        rd_mm(wa, xa, wc, xc);
        issue_odd();
        mmh(wc, xc, H1);
        ++hs;
        wait_lds();
        wait_vm<8>();
        __builtin_amdgcn_s_barrier();
        asm volatile("" ::: "memory");
        const u32x4 f = snap();
        rd_mm(wc, xc, wa, xa);
        const bool got = pick(f);
        if (wave == 0) request();
        ++tick;
        if (got) issue_even();
        young = got ? 1 : 0;
        mmh(wa, xa, H1);
        ++hs;
        wait_lds();
      }
      if (issued == hs) {
        if (deferred) mm(wc, xc);
        deferred = false;
        const unsigned long long t0 = now();
        for (unsigned idle = 1;; ++idle) {
          top();
          stage(snap());
          if (issued != hs) break;
          if (wave == 0 && (idle & 63) == 0 && give_up(self, seq, t0)) dead = true;
          __builtin_amdgcn_s_sleep(POLL_SLEEP);
        }
      }
      top();
      {
        const u32x4 f = snap();
        if (deferred) rd_mm(wa, xa, wc, xc); else rd(wa, xa, (hs & 3) * SLOT_BYTES);
        stage(f);
      }
      if (deferred) mmh(wc, xc, H1);
      wait_lds();
      __builtin_amdgcn_sched_barrier(0);
      ++hs;
      top();
      {
        const u32x4 f = snap();
        rd_mm(wc, xc, wa, xa);
        stage(f);
      }
      mmh(wa, xa, H1);
      wait_lds();
      __builtin_amdgcn_sched_barrier(0);
      ++hs;
      deferred = true;
    }
    mm(wc, xc);
    if (wave == 0) STAMP(2);
    float mx = 0.f;
#pragma unroll
    for (int t = 0; t < 8; ++t)
#pragma unroll
      for (int m = 0; m < 4; ++m)
#pragma unroll
        for (int r = 0; r < 4; ++r) mx = fmaxf(mx, fabsf(acc[t][m][r]));
    const unsigned wmx = wave_max_u32(__float_as_uint(mx));
    if (lane == 0) {
      __hip_atomic_fetch_max((lds_u32*)(smem + L_AMAX), wmx, __ATOMIC_RELAXED, __HIP_MEMORY_SCOPE_WORKGROUP);
      asm volatile("" ::: "memory");
      const unsigned cnt = __hip_atomic_fetch_add((lds_u32*)(smem + L_ACNT), 1u, __ATOMIC_RELAXED, __HIP_MEMORY_SCOPE_WORKGROUP);
      asm volatile("" ::: "memory");
      const unsigned am = __hip_atomic_load((lds_u32*)(smem + L_AMAX), __ATOMIC_RELAXED, __HIP_MEMORY_SCOPE_WORKGROUP);
      asm volatile("" ::: "memory");
      if (cnt == 7) {
        __builtin_amdgcn_raw_buffer_store_b64(u32x2{am, seq}, self, AMAX_WG + b * 8, 0, AUX_SYS);
        STAMP(9);
      }
    }
    const unsigned nearly = ti != 0 || YROWS / 32 <= (wave >> 1) * 2 ? 0 : min(2u, YROWS / 32 - (wave >> 1) * 2);
    if (nearly) {
#pragma unroll
      for (int m = 0; m < 4; ++m) {
        if ((unsigned)(m >> 1) >= nearly) break;
        const unsigned yo = EXP_Y + (((wave >> 1) * 4 + m) * 16 + tj * 2 + (wave & 1)) * 4096 + lane * 16;
#pragma unroll
        for (int r = 0; r < 4; ++r) {
          u32x4 v;
#pragma unroll
          for (int e = 0; e < 4; ++e)
            v[e] = __builtin_bit_cast(unsigned, __builtin_convertvector((f32x2{acc[2 * e][m][r], acc[2 * e + 1][m][r]}), bf16x2));
          st128<AUX_SYS>(v, self, yo + r * 1024);
        }
      }
      wait_vm<0>();
      if (lane < 16 * nearly && (lane & 7) != rank) {
        glb_u64* fp = (glb_u64*)(P.ws[lane & 7] + FY_SRC + (rank * 256 + ((wave >> 1) * 4 + (lane >> 3)) * 16 + tj * 2 + (wave & 1)) * 8);
        __hip_atomic_store(fp, pair(seq, seq), __ATOMIC_RELAXED, __HIP_MEMORY_SCOPE_SYSTEM);
      }
      if (wave == 0) STAMP(6);
    }
    if (wave == 0) {
      unsigned gm = 0, tm = 0;
      unsigned long long t0 = now();
      for (unsigned spins = 0;; ++spins) {
        asm volatile("" ::: "memory");
        u32x2 g = __builtin_amdgcn_raw_buffer_load_b64(self, AMAX_G + (b & 7) * 128 + (lane & 7) * 8, 0, AUX_SYS);
        tm = rfl(__builtin_amdgcn_raw_buffer_load_b32(self, TMO, 0, AUX_SYS));
        if (__all(g[1] >= seq)) { gm = g[0]; break; }
        if (tm == seq || ((spins & 63) == 63 && give_up(self, seq, t0))) { tm = seq; break; }
        __builtin_amdgcn_s_sleep(1);
      }
      const float amax = __uint_as_float(wave_max_u32(gm));
      float iv = amax > 0.f ? 448.f / amax : 0.f;
      if (tm == seq) iv = __uint_as_float(0x7fc00000u);
      if (lane == 0) {
        lds_st(smem, L_SCALE, __float_as_uint(iv));
        lds_st(smem, L_SFLAG, 1u);
      }
    }
    while (lds_ld(smem, L_SFLAG) == 0) __builtin_amdgcn_s_sleep(1);
    asm volatile("" ::: "memory");
    const float inv = __uint_as_float(lds_ld(smem, L_SCALE));
    if (wave == 0) STAMP(3);
    if (wave != (YROWS / 64) * 2) __builtin_amdgcn_s_sleep(OUT_DELAY);
    char* tb = smem + wave * 8192;
    const unsigned c = lane & 15, q = lane >> 4;
    const unsigned ovoff = (lane >> 3) * NDIM + (lane & 7) * 16;
    const unsigned osoff = (ti * 256 + (wave >> 1) * 64) * NDIM + rank * NBLK + tj * 256 + (wave & 1) * 128;
    u32x4 ov[8];
#pragma unroll
    for (int hf = 0; hf < 2; ++hf) {
#pragma unroll
      for (int m = 2 * hf; m < 2 * hf + 2; ++m) {
        unsigned dw[8];
#pragma unroll
        for (int r = 0; r < 4; ++r)
#pragma unroll
          for (int hh = 0; hh < 2; ++hh) {
            int p = __builtin_amdgcn_cvt_pk_fp8_f32(acc[4 * hh + 0][m][r] * inv, acc[4 * hh + 1][m][r] * inv, 0, false);
            p = __builtin_amdgcn_cvt_pk_fp8_f32(acc[4 * hh + 2][m][r] * inv, acc[4 * hh + 3][m][r] * inv, p, true);
            dw[2 * r + hh] = (unsigned)p;
          }
        const unsigned row = m * 16 + c;
        *(u32x4*)(tb + row * 128 + (((2 * q) ^ (row & 7)) * 16)) = u32x4{dw[0], dw[1], dw[2], dw[3]};
        *(u32x4*)(tb + row * 128 + (((2 * q + 1) ^ (row & 7)) * 16)) = u32x4{dw[4], dw[5], dw[6], dw[7]};
      }
#pragma unroll
      for (int it = 4 * hf; it < 4 * hf + 4; ++it) {
        const unsigned row = it * 8 + (lane >> 3);
        ov[it] = *(const u32x4*)(tb + row * 128 + (((lane & 7) ^ (row & 7)) * 16));
      }
      const unsigned chunk = (((wave >> 1) * 2 + hf) * 2 + (wave & 1)) * 128 + b;
      if ((unsigned)hf < nearly) {
      } else if (chunk - (YROWS / 16) * 128 < PUSH_O) {
        for (unsigned qq = 0; qq < 7; ++qq) {
          const unsigned p = (rank + 1 + (qq + b) % 7) & 7;
          __amdgpu_buffer_rsrc_t rp = mk_rsrc(P.out[p], OUT_BYTES);
#pragma unroll
          for (int it = 4 * hf; it < 4 * hf + 4; ++it) st128<AUX_SYS>(ov[it], rp, osoff + ovoff + it * 8 * NDIM);
        }
      } else {
#pragma unroll
        for (int it = 4 * hf; it < 4 * hf + 4; ++it)
          st128<AUX_SYS>(ov[it], self, EXP_O + chunk * 4096 + lane * 16 + (it & 3) * 1024);
      }
    }
    wait_vm<0>();
    if (lane < 2 * NR && (lane >> 3) >= nearly && (lane & 7) != rank) {
      glb_u64* fp = (glb_u64*)(P.ws[lane & 7] + FO_SRC + (rank * 2048 + (((wave >> 1) * 2 + (lane >> 3)) * 2 + (wave & 1)) * 128 + b) * 8);
      __hip_atomic_store(fp, pair(seq, seq), __ATOMIC_RELAXED, __HIP_MEMORY_SCOPE_SYSTEM);
    }
    if (wave == 0) STAMP(4);
    __amdgpu_buffer_rsrc_t ro = mk_rsrc(P.out[rank], OUT_BYTES);
#pragma unroll
    for (int it = 0; it < 8; ++it) st128<AUX_SC1>(ov[it], ro, osoff + ovoff + it * 8 * NDIM);
    for (unsigned job = b * 8 + wave; job < 7 * YROWS; job += 1024) {
      const unsigned p = (rank + 1 + job % 7) & 7, id = job / 7;
      const bool ok = wait_flag(self, FY_RDY + (p * 256 + id) * 8, seq);
      const float iv = ok ? inv : __uint_as_float(0x7fc00000u);
      unsigned dw[8];
#pragma unroll
      for (int r = 0; r < 4; ++r) {
        const u32x4 v = __builtin_amdgcn_raw_buffer_load_b128(self, STG_Y + (p * 256 + id) * 4096 + r * 1024 + lane * 16, 0, AUX_SYS);
#pragma unroll
        for (int hh = 0; hh < 2; ++hh) {
          int pk = __builtin_amdgcn_cvt_pk_fp8_f32(__uint_as_float(v[2 * hh] << 16) * iv, __uint_as_float(v[2 * hh] & 0xffff0000u) * iv, 0, false);
          pk = __builtin_amdgcn_cvt_pk_fp8_f32(__uint_as_float(v[2 * hh + 1] << 16) * iv, __uint_as_float(v[2 * hh + 1] & 0xffff0000u) * iv, pk, true);
          dw[2 * r + hh] = (unsigned)pk;
        }
      }
      *(u32x4*)(tb + c * 128 + (((2 * q) ^ (c & 7)) * 16)) = u32x4{dw[0], dw[1], dw[2], dw[3]};
      *(u32x4*)(tb + c * 128 + (((2 * q + 1) ^ (c & 7)) * 16)) = u32x4{dw[4], dw[5], dw[6], dw[7]};
#pragma unroll
      for (int it = 0; it < 2; ++it) {
        const unsigned row = it * 8 + (lane >> 3);
        const u32x4 o = *(const u32x4*)(tb + row * 128 + (((lane & 7) ^ (row & 7)) * 16));
        st128<AUX_SC1>(o, ro, ((id >> 4) * 16 + it * 8) * NDIM + p * NBLK + (id & 15) * 128 + ovoff);
      }
    }
  } else if (wave >= 1 && wave <= 3) {
    unsigned seq = 0;
    export_job(P, (wave - 1) * 128 + b, lane, self, seq, true);
    if (wave == 1) STAMP(1);
  } else if (wave == 0 && b < NPULLERS) {
    const unsigned seq = rfl(__builtin_amdgcn_raw_buffer_load_b32(self, EPOCH, 0, AUX_SYS)) + 1;
    const unsigned pq = b % 7, k = b / 7;
    const unsigned p = (rank + 1 + pq) & 7;
    const unsigned rotx = 8 * ((pq * 15 + 3) / 7);
    const unsigned voff = lane * 16;
    const u32x4 dp = mk_desc(P.ws[p], WS_NEED), dout = mk_desc(P.out[rank], OUT_BYTES);
    const unsigned ovoff = (lane >> 3) * NDIM + (lane & 7) * 16;
    const unsigned ovo[4] = {ovoff, ovoff + 8 * NDIM, ovoff + 16 * NDIM, ovoff + 24 * NDIM};
    const u32x2 seq2 = {seq, seq};
    const u32x4 nan4 = {0x7f7f7f7fu, 0x7f7f7f7fu, 0x7f7f7f7fu, 0x7f7f7f7fu};
    auto run = [&](auto phase_tag) {
      constexpr int PH = decltype(phase_tag)::value;
      constexpr unsigned J0 = PH == 0 ? 4 * PUSH_W : PUSH_O;
      constexpr unsigned NJ = PH == 0 ? 1536u + YROWS : 2048u - YROWS / 2;
      constexpr unsigned NP = PH == 0 ? NP_IN : NP_OUT;
      if (k >= NP) return;
      const unsigned tk_off = TICKET + (PH * 8 + pq) * 128;
      const unsigned tk_base = (seq - 1) * (NJ - J0 - 2 * NP) - 3 * NP - J0;
      auto decode = [&](unsigned n, unsigned& kind, unsigned& idx) {
        if (n < 72) { kind = n < 64; idx = n < 64 ? p * 512 + n : p * 1024 + n - 64; return; }
        if (n >= 1536) { kind = 2; idx = n - 1536; return; }
        const unsigned s = (n - 72) / 192, r = (n - 72) % 192, t = r / 3, e = r % 3;
        if (s < 7 && e == 2) { kind = 1; idx = (p * 8 + s + 1) * 64 + ((t + 8 * (pq + 1)) & 63); return; }
        kind = 0;
        const unsigned xi = s < 7 ? t * 2 + e : r + 8;
        if (xi < 8) { idx = (p * 8 + s + 1) * 128 + xi; return; }
        unsigned cx = xi - 8 + rotx;
        if (cx >= 120) cx -= 120;
        idx = (p * 8 + s) * 128 + 8 + cx;
      };
      auto ochunk = [&](unsigned n) -> unsigned {
        constexpr unsigned G0 = YROWS / 16, FULL = (16 - G0) * 128;
        if (n < FULL) return G0 * 128 + n;
        const unsigned g = (n - FULL) / 120, i = (n - FULL) % 120;
        return g * 128 + (i < 24 ? (i / 6) * 8 + 2 + i % 6 : i + 8);
      };
      auto f_src = [&](unsigned n) -> unsigned {
        if constexpr (PH == 1) return FO_SRC + (p * 2048 + ochunk(n)) * 8;
        unsigned kind, idx;
        decode(n, kind, idx);
        return kind == 0 ? FX_SRC + idx * 8 : kind == 1 ? FW_SRC + (idx >> 2) * 8 : FY_SRC + (p * 256 + idx) * 8;
      };
      auto addrs = [&](unsigned n, unsigned& src, unsigned& dst, unsigned& frdy) {
        if constexpr (PH == 1) {
          const unsigned ch = ochunk(n);
          const unsigned tb = ch & 127, g = tb & 7, x = tb >> 3;
          const unsigned i = (g >> 1) * 4 + (x >> 2), j = (g & 1) * 4 + (x & 3);
          src = EXP_O + ch * 4096;
          dst = (i * 256 + (ch >> 8) * 32) * NDIM + p * NBLK + j * 256 + ((ch >> 7) & 1) * 128;
          frdy = 0;
        } else {
          unsigned kind, idx;
          decode(n, kind, idx);
          if (kind == 2) {
            src = EXP_Y + idx * 4096;
            dst = STG_Y + (p * 256 + idx) * 4096;
            frdy = FY_RDY + (p * 256 + idx) * 8;
          } else {
            dst = (kind ? STG_W : STG_X) + idx * 4096;
            src = kind ? EXP_W + (rank * 512 + (idx & 511)) * 4096 : dst;
            frdy = (kind ? FW_RDY : FX_RDY) + idx * 8;
          }
        }
      };
      bool seen7 = false; (void)seen7;
      u32x4 va[4], vb[4];
      unsigned j1 = J0 + k + NP, j2 = J0 + k + 2 * NP;
      unsigned src, dst, frdy, pfrdy = DUMMY;
      bool ok = wait_flag(self, f_src(J0 + k), seq);
      STAMP(PH == 0 ? 33 : 35);
      addrs(J0 + k, src, dst, frdy);
      u32x2 fv;
      unsigned dummy = 0;
      a_ld4(va, voff, dp, src);
      a_ld2(fv, ds, f_src(j1));
      unsigned nsrc, ndst, nfrdy;
      addrs(j1, nsrc, ndst, nfrdy);
      a_wait<0>(va, dummy, fv);
      auto step = [&](u32x4 (&cur)[4], u32x4 (&nxt)[4]) -> bool {
        if (j1 >= NJ) return false;
        const bool cur_ok = ok;
        if (rfl(min(fv[0], fv[1])) < seq) ok = wait_flag(self, f_src(j1), seq) && ok;
        a_ld4(nxt, voff, dp, nsrc);
        unsigned tk = 1;
        const bool has2 = j2 < NJ;
        if (has2) a_ticket_flag(tk, fv, ds, tk_off, f_src(j2));
        const unsigned cdst = dst, cpf = pfrdy;
        pfrdy = frdy; dst = ndst; frdy = nfrdy;
        if constexpr (PH == 0) {
          a_st4_sc1(cur, voff, ds, cdst);
        } else {
          if (!cur_ok) { cur[0] = nan4; cur[1] = nan4; cur[2] = nan4; cur[3] = nan4; }
          a_st4_rows(cur, ovo, dout, cdst);
        }
        j1 = j2;
        addrs(has2 ? j1 : J0, nsrc, ndst, nfrdy);
        asm volatile("" : "+s"(nsrc), "+s"(ndst), "+s"(nfrdy));
        if constexpr (PH == 0) {
          a_wait<0>(nxt, tk, fv);
          a_st2_sc1(seq2, ds, pfrdy);
        } else {
          a_wait<4>(nxt, tk, fv);
        }
        (void)cpf;
        j2 = has2 ? rfl(tk) - tk_base : NJ;
        if (j2 > NJ) j2 = NJ;
        return true;
      };
      auto last = [&](u32x4 (&cur)[4]) {
        if constexpr (PH == 0) {
          a_st4_sc1(cur, voff, ds, dst);
          wait_vm<0>();
          a_st2_sc1(seq2, ds, frdy);
        } else {
          if (!ok) { cur[0] = nan4; cur[1] = nan4; cur[2] = nan4; cur[3] = nan4; }
          a_st4_rows(cur, ovo, dout, dst);
        }
      };
      for (;;) {
        if (!step(va, vb)) { last(va); break; }
        if (!step(vb, va)) { last(vb); break; }
      }
    };
    STAMP(32);
    run(std::integral_constant<int, 0>{});
    STAMP(34);
    run(std::integral_constant<int, 1>{});
    if (PUSH_O > 0 && k == 0) {
      if (!wait_flag(self, FO_SRC + (p * 2048 + (YROWS / 16) * 128 + (lane % (PUSH_O > 0 ? PUSH_O : 1))) * 8, seq) && lane == 0)
        *(unsigned*)(P.out[rank] + YROWS * NDIM + p * NBLK) = 0x7f7f7f7fu;
    }
    STAMP(36);
  } else if (wave == 0 && b == REDUCER) {
    const unsigned seq = rfl(__builtin_amdgcn_raw_buffer_load_b32(self, EPOCH, 0, AUX_SYS)) + 1;
    unsigned gm = 0;
    unsigned long long t0 = now();
    for (unsigned spins = 0;; ++spins) {
      asm volatile("" ::: "memory");
      u32x4 a = __builtin_amdgcn_raw_buffer_load_b128(self, AMAX_WG + lane * 16, 0, AUX_SYS);
      gm = max(a[0], a[2]);
      if (__all(a[1] >= seq && a[3] >= seq)) break;
      if ((spins & 63) == 63 && give_up(self, seq, t0)) break;
    }
    gm = wave_max_u32(gm);
    glb_u64* gp = (glb_u64*)(P.ws[lane >> 3] + AMAX_G + (lane & 7) * 128 + rank * 8);
    __hip_atomic_store(gp, pair(gm, seq), __ATOMIC_RELAXED, __HIP_MEMORY_SCOPE_SYSTEM);
    if (lane == 0) __builtin_amdgcn_raw_buffer_store_b32(seq, self, EPOCH, 0, AUX_SYS);
    STAMP(39);
  }
}

}

extern "C" void kernel_launch(int rank, int nranks, ncclComm_t comm, hipStream_t stream, void* const* d_in,
                              const int* in_sizes, int n_in, void* d_out, int out_size, const int* in_dtypes,
                              int out_dtype, void* const* d_out_all, void* const* d_ws_all, size_t ws_bytes) {
  (void)comm; (void)in_sizes; (void)in_dtypes; (void)out_dtype; (void)out_size;
  if (nranks != NR || n_in < 2 || (unsigned)rank >= (unsigned)NR || ws_bytes < WS_NEED) {
    fprintf(stderr, "kernel_launch: unsupported configuration\n");
    return;
  }
  Params P;
  P.x = (const char*)d_in[0];
  P.w = (const char*)d_in[1];
  for (int p = 0; p < NR; ++p) {
    P.out[p] = (char*)d_out_all[p];
    P.ws[p] = (char*)d_ws_all[p];
  }
  P.out[rank] = (char*)d_out;
  P.rank = rank;
  hipLaunchKernelGGL(fused_kernel, dim3(NWG), dim3(NTHREADS), 0, stream, P);
  hipError_t e = hipGetLastError();
  if (e != hipSuccess) fprintf(stderr, "launch failed: %s\n", hipGetErrorString(e));
}
```

```cpp
#include <hip/hip_runtime.h>
#include <rccl/rccl.h>
#include <cstdio>
#include <cstdint>
#include <type_traits>

namespace {

constexpr int NR = 8;
constexpr int MDIM = 4096, KSH = 512, NDIM = 16384, NBLK = 2048;
constexpr int NWG = 256;
constexpr int NTHREADS = 512;
#define NP_IN 16
#define NP_OUT 16
constexpr int NPULLERS = 7 * (NP_IN > NP_OUT ? NP_IN : NP_OUT);
constexpr int REDUCER = 127;
static_assert(NPULLERS <= REDUCER, "too many pullers");
#define PUSH_W 12
#define PACE_W 3
#define YROWS 160
#define POLL_SLEEP 4
#define EXPORT_DELAY 16
#define OUT_DELAY 8
constexpr unsigned long long DEADLINE_TICKS = 100000000ull;

constexpr unsigned STG_X = 0;
constexpr unsigned STG_W = 32u << 20;
constexpr unsigned EXP_W = 48u << 20;
constexpr unsigned EXP_O = 64u << 20;
constexpr unsigned FLG = 72u << 20;
constexpr unsigned FX_SRC = FLG;
constexpr unsigned FX_RDY = FLG + (64u << 10);
constexpr unsigned FW_SRC = FLG + (128u << 10);
constexpr unsigned FW_RDY = FLG + (144u << 10);
constexpr unsigned FO_SRC = FLG + (192u << 10);
constexpr unsigned AMAX_WG = FLG + (320u << 10);
constexpr unsigned AMAX_G = FLG + (324u << 10);
constexpr unsigned EPOCH = FLG + (328u << 10);
constexpr unsigned TMO = EPOCH + 256;
constexpr unsigned TICKET = FLG + (332u << 10);
constexpr unsigned FY_SRC = FLG + (336u << 10);
constexpr unsigned FY_RDY = FLG + (352u << 10);
constexpr unsigned DBG = FLG + (512u << 10);
constexpr unsigned EXP_Y = FLG + (1u << 20);
constexpr unsigned STG_Y = FLG + (2u << 20);
constexpr unsigned WS_NEED = FLG + (10u << 20);

constexpr int SLOT_BYTES = 32 * 1024;
constexpr int L_CTRL = 4 * SLOT_BYTES;
constexpr int L_AMAX = L_CTRL;
constexpr int L_ACNT = L_CTRL + 4;
constexpr int L_SCALE = L_CTRL + 8;
constexpr int L_SFLAG = L_CTRL + 12;
constexpr int L_FLG = L_CTRL + 64;
constexpr int LDS_BYTES = L_FLG + 2048;

typedef __attribute__((ext_vector_type(8))) short bf16x8;
typedef __attribute__((ext_vector_type(4))) float f32x4;
typedef __attribute__((ext_vector_type(4))) unsigned u32x4;
typedef __attribute__((ext_vector_type(2))) unsigned u32x2;
typedef __attribute__((ext_vector_type(2))) float f32x2;
typedef __attribute__((ext_vector_type(2))) __bf16 bf16x2;
typedef __attribute__((address_space(3))) unsigned lds_u32;
typedef __attribute__((address_space(1))) unsigned glb_u32;
typedef __attribute__((address_space(1))) unsigned long long glb_u64;

struct Params {
  const char* x;
  const char* w;
  char* out;
  char* ws[NR];
  int rank;
};

constexpr int AUX_SC1 = 16, AUX_SYS = 17;
constexpr unsigned OUT_BYTES = (unsigned)MDIM * NDIM;

__device__ __forceinline__ __amdgpu_buffer_rsrc_t mk_rsrc(const void* p, unsigned bytes) {
  return __builtin_amdgcn_make_buffer_rsrc(const_cast<void*>(p), 0, bytes, 0x00020000);
}
template <int AUX> __device__ __forceinline__ void st128(u32x4 v, __amdgpu_buffer_rsrc_t r, unsigned off) {
  __builtin_amdgcn_raw_buffer_store_b128(v, r, off, 0, AUX);
}
template <int N> __device__ __forceinline__ void wait_vm() { asm volatile("s_waitcnt vmcnt(%0)" ::"i"(N) : "memory"); }
__device__ __forceinline__ void wait_lds() {
  asm volatile("" ::: "memory");
  __builtin_amdgcn_s_waitcnt(0xC07F);
  asm volatile("" ::: "memory");
}
__device__ __forceinline__ unsigned rfl(unsigned v) { return __builtin_amdgcn_readfirstlane(v); }
__device__ __forceinline__ unsigned lds_ld(const char* smem, int off) {
  return rfl(__hip_atomic_load((const lds_u32*)(smem + off), __ATOMIC_RELAXED, __HIP_MEMORY_SCOPE_WORKGROUP));
}
__device__ __forceinline__ void lds_st(char* smem, int off, unsigned v) {
  __hip_atomic_store((lds_u32*)(smem + off), v, __ATOMIC_RELAXED, __HIP_MEMORY_SCOPE_WORKGROUP);
}
__device__ __forceinline__ unsigned wave_max_u32(unsigned v) {
  v = max(v, (unsigned)__builtin_amdgcn_update_dpp(0, (int)v, 0x111, 0xf, 0xf, false));
  v = max(v, (unsigned)__builtin_amdgcn_update_dpp(0, (int)v, 0x112, 0xf, 0xf, false));
  v = max(v, (unsigned)__builtin_amdgcn_update_dpp(0, (int)v, 0x114, 0xf, 0xf, false));
  v = max(v, (unsigned)__builtin_amdgcn_update_dpp(0, (int)v, 0x118, 0xf, 0xf, false));
  const unsigned a = __builtin_amdgcn_readlane(v, 15), b = __builtin_amdgcn_readlane(v, 31);
  const unsigned c = __builtin_amdgcn_readlane(v, 47), d = __builtin_amdgcn_readlane(v, 63);
  return max(max(a, b), max(c, d));
}
__device__ __forceinline__ unsigned long long now() { return __builtin_amdgcn_s_memrealtime(); }
__device__ __forceinline__ unsigned long long pair(unsigned lo, unsigned hi) { return ((unsigned long long)hi << 32) | lo; }

#define STAMP(slot) do {} while (0)

__device__ __forceinline__ bool give_up(__amdgpu_buffer_rsrc_t self, unsigned seq, unsigned long long t0) {
  unsigned t = rfl(__builtin_amdgcn_raw_buffer_load_b32(self, TMO, 0, AUX_SYS));
  if (t == seq || now() - t0 > DEADLINE_TICKS) {
    __builtin_amdgcn_raw_buffer_store_b32(seq, self, TMO, 0, AUX_SYS);
    return true;
  }
  return false;
}

__device__ __forceinline__ bool wait_flag(__amdgpu_buffer_rsrc_t self, unsigned off, unsigned seq) {
  unsigned long long t0 = now();
  for (unsigned spins = 0;; ++spins) {
    asm volatile("" ::: "memory");
    u32x2 v = __builtin_amdgcn_raw_buffer_load_b64(self, off, 0, AUX_SYS);
    if (__all(v[0] >= seq && v[1] >= seq)) return true;
    if ((spins & 63) == 63 && give_up(self, seq, t0)) return false;
    __builtin_amdgcn_s_sleep(1);
  }
}

__device__ __forceinline__ u32x4 mk_desc(const void* p, unsigned bytes) {
  const unsigned long long a = (unsigned long long)p;
  return u32x4{(unsigned)a, (unsigned)(a >> 32) & 0xffffu, bytes, 0x00020000u};
}
template <int H>
__device__ __forceinline__ void glds22(unsigned voff, const char* wsrc, const char* xsrc, unsigned lw, unsigned lx) {
  unsigned keep;
  asm volatile(
      "s_mov_b32 %0, m0\n\ts_mov_b32 m0, %4\n\ts_nop 0\n\t"
      "global_load_lds_dwordx4 %1, %2 offset:%6 sc1\n\t"
      "global_load_lds_dwordx4 %1, %2 offset:%7 sc1\n\t"
      "s_mov_b32 m0, %5\n\ts_nop 0\n\t"
      "global_load_lds_dwordx4 %1, %3 offset:%6 sc1\n\t"
      "global_load_lds_dwordx4 %1, %3 offset:%7 sc1\n\t"
      "s_mov_b32 m0, %0"
      : "=&s"(keep) : "v"(voff), "s"(wsrc), "s"(xsrc), "s"(lw), "s"(lx), "i"(H * 2048), "i"(H * 2048 + 1024) : "memory");
}
__device__ __forceinline__ void a_ld4(u32x4 (&d)[4], unsigned voff, u32x4 rs, unsigned soff) {
  asm volatile(
      "buffer_load_dwordx4 %0, %4, %5, %6 offen sc0 sc1\n\t"
      "buffer_load_dwordx4 %1, %4, %5, %6 offen offset:1024 sc0 sc1\n\t"
      "buffer_load_dwordx4 %2, %4, %5, %6 offen offset:2048 sc0 sc1\n\t"
      "buffer_load_dwordx4 %3, %4, %5, %6 offen offset:3072 sc0 sc1"
      : "=&v"(d[0]), "=&v"(d[1]), "=&v"(d[2]), "=&v"(d[3]) : "v"(voff), "s"(rs), "s"(soff) : "memory");
}
__device__ __forceinline__ void a_st4_sc1(const u32x4 (&d)[4], unsigned voff, u32x4 rs, unsigned soff) {
  asm volatile(
      "buffer_store_dwordx4 %0, %4, %5, %6 offen sc1\n\t"
      "buffer_store_dwordx4 %1, %4, %5, %6 offen offset:1024 sc1\n\t"
      "buffer_store_dwordx4 %2, %4, %5, %6 offen offset:2048 sc1\n\t"
      "buffer_store_dwordx4 %3, %4, %5, %6 offen offset:3072 sc1\n\ts_nop 1"
      :: "v"(d[0]), "v"(d[1]), "v"(d[2]), "v"(d[3]), "v"(voff), "s"(rs), "s"(soff) : "memory");
}
__device__ __forceinline__ void a_st4_rows(const u32x4 (&d)[4], const unsigned (&vo)[4], u32x4 rs, unsigned soff) {
  asm volatile(
      "buffer_store_dwordx4 %0, %4, %8, %9 offen sc1\n\t"
      "buffer_store_dwordx4 %1, %5, %8, %9 offen sc1\n\t"
      "buffer_store_dwordx4 %2, %6, %8, %9 offen sc1\n\t"
      "buffer_store_dwordx4 %3, %7, %8, %9 offen sc1\n\ts_nop 1"
      :: "v"(d[0]), "v"(d[1]), "v"(d[2]), "v"(d[3]), "v"(vo[0]), "v"(vo[1]), "v"(vo[2]), "v"(vo[3]), "s"(rs), "s"(soff) : "memory");
}
__device__ __forceinline__ void a_ld2(u32x2& d, u32x4 rs, unsigned soff) {
  asm volatile("buffer_load_dwordx2 %0, off, %1, %2 sc0 sc1" : "=&v"(d) : "s"(rs), "s"(soff) : "memory");
}
__device__ __forceinline__ void glds_flag(unsigned voff, const char* base, unsigned l) {
  unsigned keep;
  asm volatile(
      "s_mov_b32 %0, m0\n\ts_mov_b32 m0, %3\n\ts_nop 0\n\t"
      "global_load_lds_dwordx4 %1, %2 sc0 sc1\n\t"
      "s_mov_b32 m0, %0"
      : "=&s"(keep) : "v"(voff), "s"(base), "s"(l) : "memory");
}
__device__ __forceinline__ void a_st2_sc1(u32x2 v, u32x4 rs, unsigned soff) {
  asm volatile("buffer_store_dwordx2 %0, off, %1, %2 sc1" :: "v"(v), "s"(rs), "s"(soff) : "memory");
}
__device__ __forceinline__ void a_ticket_flag(unsigned& tk, u32x2& fv, u32x4 rs, unsigned tk_off, unsigned f_off) {
  asm volatile(
      "s_mov_b64 exec, 1\n\t"
      "buffer_atomic_add %0, off, %2, %3 sc0\n\t"
      "s_mov_b64 exec, -1\n\t"
      "buffer_load_dwordx2 %1, off, %2, %4 sc0 sc1"
      : "+v"(tk), "+v"(fv) : "s"(rs), "s"(tk_off), "s"(f_off) : "memory");
}
template <int N> __device__ __forceinline__ void a_wait(u32x4 (&d)[4], unsigned& a, u32x2& b) {
  asm volatile("s_waitcnt vmcnt(%6)" : "+v"(d[0]), "+v"(d[1]), "+v"(d[2]), "+v"(d[3]), "+v"(a), "+v"(b) : "i"(N) : "memory");
}

__device__ __forceinline__ void export_job(const Params& P, unsigned n, unsigned lane, __amdgpu_buffer_rsrc_t self,
                                           unsigned& seq, bool first) {
  const unsigned rank = P.rank;
  const unsigned s = n / 384, r = n % 384;
  const unsigned sg = rank * 8 + s;
  if (r < 256) {
    const unsigned d = r & 7, j = r >> 4, h = (r >> 3) & 1;
    __amdgpu_buffer_rsrc_t rw = mk_rsrc(P.w, KSH * NDIM * 2);
    const unsigned o = lane & 15, g = lane >> 4;
    const unsigned voff = ((g * 8) * NDIM + o * 8) * 2;
    const unsigned soff = ((s * 64 + h * 32) * NDIM + d * NBLK + j * 128) * 2;
    u32x4 row[8];
#pragma unroll
    for (int jj = 0; jj < 8; ++jj) row[jj] = __builtin_amdgcn_raw_buffer_load_b128(rw, voff + jj * NDIM * 2, soff, 0);
    if (first) seq = rfl(__builtin_amdgcn_raw_buffer_load_b32(self, EPOCH, 0, AUX_SYS)) + 1;
    const bool direct = d == rank || (s == 0 && j < PUSH_W);
    char* const base = P.ws[direct ? d : rank];
    __amdgpu_buffer_rsrc_t rd = mk_rsrc(base, WS_NEED);
    const unsigned dst = (direct ? STG_W + sg * 16 * 16384 : EXP_W + (d * 8 + s) * 16 * 16384) + j * 16384 + h * 2048;
#pragma unroll
    for (int t = 0; t < 8; ++t) {
      u32x4 f;
#pragma unroll
      for (int e = 0; e < 4; ++e)
        f[e] = __builtin_amdgcn_perm(row[2 * e + 1][t >> 1], row[2 * e][t >> 1], (t & 1) ? 0x07060302u : 0x05040100u);
      st128<AUX_SYS>(f, rd, dst + lane * 16 + (t & 1) * 1024 + (t >> 1) * 4096);
      if (direct && d != rank) wait_vm<PACE_W - 1>();
    }
    wait_vm<0>();
    if (direct) {
      if (lane < 4) __builtin_amdgcn_raw_buffer_store_b32(seq, rd, FW_RDY + ((sg * 16 + j) * 4 + lane) * 8 + h * 4, 0, AUX_SYS);
    } else if (lane == 0) {
      __amdgpu_buffer_rsrc_t rp = mk_rsrc(P.ws[d], WS_NEED);
      __builtin_amdgcn_raw_buffer_store_b32(seq, rp, FW_SRC + (sg * 16 + j) * 8 + h * 4, 0, AUX_SYS);
    }
  } else {
    const unsigned i = (r - 256) >> 3, sub = (r - 256) & 7;
    __amdgpu_buffer_rsrc_t rx = mk_rsrc(P.x, MDIM * KSH * 2);
    const unsigned c = lane & 15, q = lane >> 4;
    const unsigned voff = (c * KSH + q * 8) * 2;
    const unsigned soff = ((i * 256 + sub * 32) * KSH + s * 64) * 2;
    u32x4 f[4];
#pragma unroll
    for (int b = 0; b < 4; ++b)
      f[b] = __builtin_amdgcn_raw_buffer_load_b128(rx, voff + ((b & 1) * 16 * KSH + (b >> 1) * 32) * 2, soff, 0);
    if (first) seq = rfl(__builtin_amdgcn_raw_buffer_load_b32(self, EPOCH, 0, AUX_SYS)) + 1;
    const unsigned idx = (sg * 16 + i) * 8 + sub;
#pragma unroll
    for (int b = 0; b < 4; ++b) st128<AUX_SYS>(f[b], self, STG_X + idx * 4096 + lane * 16 + b * 1024);
    wait_vm<0>();
    if (lane < NR) {
      glb_u64* fp = (glb_u64*)(P.ws[lane] + (lane == rank ? FX_RDY : FX_SRC) + idx * 8);
      __hip_atomic_store(fp, pair(seq, seq), __ATOMIC_RELAXED, __HIP_MEMORY_SCOPE_SYSTEM);
    }
  }
}

__global__ void __launch_bounds__(NTHREADS, 2) fused_kernel(Params P) {
  __shared__ __attribute__((aligned(16))) unsigned smem32[LDS_BYTES / 4];
  char* const smem = (char*)smem32;
  const unsigned lane = threadIdx.x & 63;
  const unsigned wave = rfl(threadIdx.x >> 6);
  const unsigned rank = P.rank;
  char* const wsl = P.ws[rank];
  __amdgpu_buffer_rsrc_t self = mk_rsrc(wsl, WS_NEED);
  const u32x4 ds = mk_desc(wsl, WS_NEED);
  const bool is_gemm = blockIdx.x >= 128;
  const unsigned b = blockIdx.x & 127;

  if (is_gemm) {
    const unsigned xg = b & 7, xi = b >> 3;
    const unsigned ti = (xg >> 1) * 4 + (xi >> 2), tj = (xg & 1) * 4 + (xi & 3);
    if (threadIdx.x < 16) smem32[L_CTRL / 4 + threadIdx.x] = 0;
    STAMP(wave == 0 ? 0 : 63);
    unsigned seq = 0;
    __builtin_amdgcn_s_sleep(EXPORT_DELAY);
    export_job(P, 384 + wave * 128 + b, lane, self, seq, true);
    export_job(P, 384 + 1024 + wave * 128 + b, lane, self, seq, false);
    if (wave < 5) export_job(P, 384 + 2048 + wave * 128 + b, lane, self, seq, false);
    if (wave == 0) STAMP(5);

    f32x4 acc[8][4];
#pragma unroll
    for (int t = 0; t < 8; ++t)
#pragma unroll
      for (int m = 0; m < 4; ++m) acc[t][m] = f32x4{0.f, 0.f, 0.f, 0.f};
    const unsigned lds0 = (unsigned)(uintptr_t)(lds_u32*)smem32 + wave * 2048;
    const unsigned voff = lane * 16;
    const char* xsrc = wsl + STG_X + (ti * 8 + wave) * 4096;
    const char* wsrc = wsl + STG_W + (tj * 8 + wave) * 4096;
    unsigned sp = 0;
    const unsigned fbase = (lane & 4) ? FW_RDY + ((lane >> 3) * 512 + tj * 8) * 8 + (lane & 3) * 16
                                      : FX_RDY + ((lane >> 3) * 1024 + ti * 8) * 8 + (lane & 3) * 16;
    const unsigned fsh = (lane & 4) ? 9 : 10;
    const unsigned lpad = (unsigned)(uintptr_t)(lds_u32*)smem32 + L_FLG;
    const unsigned lpadv = lpad + lane * 16;
    unsigned tick = 0;
    bool dead = false;
    auto request = [&]() {
      if (!dead) {
        glds_flag(fbase + ((sp & 7) << fsh), wsl, lpad + (tick & 1) * 1024);
      } else {
        const u32x4 all = {seq, seq, seq, seq};
        asm volatile("ds_write_b128 %0, %1\n\ts_waitcnt lgkmcnt(0)" :: "v"(lpadv + (tick & 1) * 1024), "v"(all) : "memory");
      }
    };
    unsigned issued = 0;
    const char* wcur = wsrc;
    const char* xcur = xsrc;
    const char* const padp = smem + L_FLG + lane * 16;
    auto snap = [&]() -> u32x4 { return *(const u32x4*)(padp + ((tick + 1) & 1) * 1024); };
    auto pick = [&](const u32x4 f) -> bool {
      unsigned long long m = __ballot(min(min(f[0], f[1]), min(f[2], f[3])) >= seq && sp < 8);
      m &= m >> 4; m &= m >> 2; m &= m >> 1; m &= 0x0101010101010101ull;
      if (m == 0) return false;
      const unsigned l8 = __builtin_ctzll(m);
      const unsigned sg = l8 + __builtin_amdgcn_readlane(sp, l8);
      sp += (lane >> 3) == (l8 >> 3) ? 1 : 0;
      wcur = wsrc + (size_t)sg * (64 * 4096);
      xcur = xsrc + (size_t)sg * (128 * 4096);
      return true;
    };
    auto issue_even = [&]() {
      const unsigned l = lds0 + (issued & 2) * SLOT_BYTES;
      glds22<0>(voff, wcur, xcur, l, l + 16384);
      ++issued;
    };
    auto issue_odd = [&]() {
      const unsigned l = lds0 + (issued & 2) * SLOT_BYTES + SLOT_BYTES - 2048;
      glds22<1>(voff, wcur, xcur, l, l + 16384);
      ++issued;
    };
    const char* wb = smem + (wave & 1) * 8192 + lane * 16;
    const char* xb = smem + 16384 + (wave >> 1) * 4096 + lane * 16;
    bf16x8 wa[8], xa[4], wc[8], xc[4];
    auto mmh = [&](bf16x8 (&wf)[8], bf16x8 (&xf)[4], auto h_tag) {
      constexpr int H = decltype(h_tag)::value;
      __builtin_amdgcn_sched_barrier(0);
#pragma unroll
      for (int t = 4 * H; t < 4 * H + 4; ++t)
#pragma unroll
        for (int m = 0; m < 4; ++m) acc[t][m] = __builtin_amdgcn_mfma_f32_16x16x32_bf16(wf[t], xf[m], acc[t][m], 0, 0, 0);
      __builtin_amdgcn_sched_barrier(0);
    };
    constexpr std::integral_constant<int, 0> H0{};
    constexpr std::integral_constant<int, 1> H1{};
    auto mm = [&](bf16x8 (&wf)[8], bf16x8 (&xf)[4]) { mmh(wf, xf, H0); mmh(wf, xf, H1); };
    auto rd = [&](bf16x8 (&wf)[8], bf16x8 (&xf)[4], unsigned so) {
#pragma unroll
      for (int m = 0; m < 4; ++m) xf[m] = *(const bf16x8*)(xb + so + m * 1024);
#pragma unroll
      for (int t = 0; t < 8; ++t) wf[t] = *(const bf16x8*)(wb + so + t * 1024);
    };
    unsigned hs = 0, young = 0;
    auto top = [&]() {
      const unsigned ahead = issued - hs;
      unsigned allow = ahead > 1 ? ahead - 1 : 0;
      if (wave == 0 && young < allow && issued < 127) allow = young;
      if (allow == 0) wait_vm<0>(); else if (allow == 1) wait_vm<4>(); else wait_vm<8>();
      __builtin_amdgcn_s_barrier();
      asm volatile("" ::: "memory");
    };
    auto stage = [&](const u32x4 f) {
      const unsigned lim = hs + 3 < 127 ? hs + 3 : 127, before = issued;
      const char* wodd = wcur;
      const char* xodd = xcur;
      bool got = false;
      if (issued + (issued & 1) <= lim && tick) got = pick(f);
      if (wave == 0 && issued + (got ? 2 : 0) < 127) request();
      if ((issued & 1) && issued <= lim) {
        const unsigned l = lds0 + (issued & 2) * SLOT_BYTES + SLOT_BYTES - 2048;
        glds22<1>(voff, wodd, xodd, l, l + 16384);
        ++issued;
      }
      if (got) {
        issue_even();
        if (issued <= lim) issue_odd();
      }
      young = issued - before;
      ++tick;
    };
    auto rd_mm = [&](bf16x8 (&wf)[8], bf16x8 (&xf)[4], bf16x8 (&wp)[8], bf16x8 (&xp)[4]) {
      __builtin_amdgcn_sched_barrier(0);
      rd(wf, xf, (hs & 3) * SLOT_BYTES);
#pragma unroll
      for (int t = 0; t < 4; ++t)
#pragma unroll
        for (int m = 0; m < 4; ++m) acc[t][m] = __builtin_amdgcn_mfma_f32_16x16x32_bf16(wp[t], xp[m], acc[t][m], 0, 0, 0);
#pragma unroll
      for (int g = 0; g < 4; ++g) {
        __builtin_amdgcn_sched_group_barrier(0x008, 1, 0);
        __builtin_amdgcn_sched_group_barrier(0x100, 1, 0);
        __builtin_amdgcn_sched_group_barrier(0x008, 1, 0);
        __builtin_amdgcn_sched_group_barrier(0x100, 1, 0);
        __builtin_amdgcn_sched_group_barrier(0x008, 1, 0);
        __builtin_amdgcn_sched_group_barrier(0x100, 1, 0);
        __builtin_amdgcn_sched_group_barrier(0x008, 1, 0);
      }
      __builtin_amdgcn_sched_barrier(0);
    };
    bool deferred = false;
    while (hs < 128) {
      while (issued == hs + 3 && deferred && young == 1 && hs < 124) {
        wait_vm<8>();
        __builtin_amdgcn_s_barrier();
        asm volatile("" ::: "memory");
        rd_mm(wa, xa, wc, xc);
        issue_odd();
        mmh(wc, xc, H1);
        ++hs;
        wait_lds();
        wait_vm<8>();
        __builtin_amdgcn_s_barrier();
        asm volatile("" ::: "memory");
        const u32x4 f = snap();
        rd_mm(wc, xc, wa, xa);
        const bool got = pick(f);
        if (wave == 0) request();
        ++tick;
        if (got) issue_even();
        young = got ? 1 : 0;
        mmh(wa, xa, H1);
        ++hs;
        wait_lds();
      }
      if (issued == hs) {
        if (deferred) mm(wc, xc);
        deferred = false;
        const unsigned long long t0 = now();
        for (unsigned idle = 1;; ++idle) {
          top();
          stage(snap());
          if (issued != hs) break;
          if (wave == 0 && (idle & 63) == 0 && give_up(self, seq, t0)) dead = true;
          __builtin_amdgcn_s_sleep(POLL_SLEEP);
        }
      }
      top();
      {
        const u32x4 f = snap();
        if (deferred) rd_mm(wa, xa, wc, xc); else rd(wa, xa, (hs & 3) * SLOT_BYTES);
        stage(f);
      }
      if (deferred) mmh(wc, xc, H1);
      wait_lds();
      __builtin_amdgcn_sched_barrier(0);
      ++hs;
      top();
      {
        const u32x4 f = snap();
        rd_mm(wc, xc, wa, xa);
        stage(f);
      }
      mmh(wa, xa, H1);
      wait_lds();
      __builtin_amdgcn_sched_barrier(0);
      ++hs;
      deferred = true;
    }
    mm(wc, xc);
    if (wave == 0) STAMP(2);
    float mx = 0.f;
#pragma unroll
    for (int t = 0; t < 8; ++t)
#pragma unroll
      for (int m = 0; m < 4; ++m)
#pragma unroll
        for (int r = 0; r < 4; ++r) mx = fmaxf(mx, fabsf(acc[t][m][r]));
    const unsigned wmx = wave_max_u32(__float_as_uint(mx));
    if (lane == 0) {
      __hip_atomic_fetch_max((lds_u32*)(smem + L_AMAX), wmx, __ATOMIC_RELAXED, __HIP_MEMORY_SCOPE_WORKGROUP);
      asm volatile("" ::: "memory");
      const unsigned cnt = __hip_atomic_fetch_add((lds_u32*)(smem + L_ACNT), 1u, __ATOMIC_RELAXED, __HIP_MEMORY_SCOPE_WORKGROUP);
      asm volatile("" ::: "memory");
      const unsigned am = __hip_atomic_load((lds_u32*)(smem + L_AMAX), __ATOMIC_RELAXED, __HIP_MEMORY_SCOPE_WORKGROUP);
      asm volatile("" ::: "memory");
      if (cnt == 7) {
        __builtin_amdgcn_raw_buffer_store_b64(u32x2{am, seq}, self, AMAX_WG + b * 8, 0, AUX_SYS);
        STAMP(9);
      }
    }
    const unsigned nearly = ti != 0 || YROWS / 32 <= (wave >> 1) * 2 ? 0 : min(2u, YROWS / 32 - (wave >> 1) * 2);
    if (nearly) {
#pragma unroll
      for (int m = 0; m < 4; ++m) {
        if ((unsigned)(m >> 1) >= nearly) break;
        const unsigned yo = EXP_Y + (((wave >> 1) * 4 + m) * 16 + tj * 2 + (wave & 1)) * 4096 + lane * 16;
#pragma unroll
        for (int r = 0; r < 4; ++r) {
          u32x4 v;
#pragma unroll
          for (int e = 0; e < 4; ++e)
            v[e] = __builtin_bit_cast(unsigned, __builtin_convertvector((f32x2{acc[2 * e][m][r], acc[2 * e + 1][m][r]}), bf16x2));
          st128<AUX_SYS>(v, self, yo + r * 1024);
        }
      }
      wait_vm<0>();
      if (lane < 16 * nearly && (lane & 7) != rank) {
        glb_u64* fp = (glb_u64*)(P.ws[lane & 7] + FY_SRC + (rank * 256 + ((wave >> 1) * 4 + (lane >> 3)) * 16 + tj * 2 + (wave & 1)) * 8);
        __hip_atomic_store(fp, pair(seq, seq), __ATOMIC_RELAXED, __HIP_MEMORY_SCOPE_SYSTEM);
      }
      if (wave == 0) STAMP(6);
    }
    if (wave == 0) {
      unsigned gm = 0, tm = 0;
      unsigned long long t0 = now();
      for (unsigned spins = 0;; ++spins) {
        asm volatile("" ::: "memory");
        u32x2 g = __builtin_amdgcn_raw_buffer_load_b64(self, AMAX_G + (b & 7) * 128 + (lane & 7) * 8, 0, AUX_SYS);
        tm = rfl(__builtin_amdgcn_raw_buffer_load_b32(self, TMO, 0, AUX_SYS));
        if (__all(g[1] >= seq)) { gm = g[0]; break; }
        if (tm == seq || ((spins & 63) == 63 && give_up(self, seq, t0))) { tm = seq; break; }
        __builtin_amdgcn_s_sleep(1);
      }
      const float amax = __uint_as_float(wave_max_u32(gm));
      float iv = amax > 0.f ? 448.f / amax : 0.f;
      if (tm == seq) iv = __uint_as_float(0x7fc00000u);
      if (lane == 0) {
        lds_st(smem, L_SCALE, __float_as_uint(iv));
        lds_st(smem, L_SFLAG, 1u);
      }
    }
    while (lds_ld(smem, L_SFLAG) == 0) __builtin_amdgcn_s_sleep(1);
    asm volatile("" ::: "memory");
    const float inv = __uint_as_float(lds_ld(smem, L_SCALE));
    if (wave == 0) STAMP(3);
    if (wave != (YROWS / 64) * 2) __builtin_amdgcn_s_sleep(OUT_DELAY);
    char* tb = smem + wave * 8192;
    const unsigned c = lane & 15, q = lane >> 4;
    const unsigned ovoff = (lane >> 3) * NDIM + (lane & 7) * 16;
    const unsigned osoff = (ti * 256 + (wave >> 1) * 64) * NDIM + rank * NBLK + tj * 256 + (wave & 1) * 128;
    u32x4 ov[8];
#pragma unroll
    for (int hf = 0; hf < 2; ++hf) {
#pragma unroll
      for (int m = 2 * hf; m < 2 * hf + 2; ++m) {
        unsigned dw[8];
#pragma unroll
        for (int r = 0; r < 4; ++r)
#pragma unroll
          for (int hh = 0; hh < 2; ++hh) {
            int p = __builtin_amdgcn_cvt_pk_fp8_f32(acc[4 * hh + 0][m][r] * inv, acc[4 * hh + 1][m][r] * inv, 0, false);
            p = __builtin_amdgcn_cvt_pk_fp8_f32(acc[4 * hh + 2][m][r] * inv, acc[4 * hh + 3][m][r] * inv, p, true);
            dw[2 * r + hh] = (unsigned)p;
          }
        const unsigned row = m * 16 + c;
        *(u32x4*)(tb + row * 128 + (((2 * q) ^ (row & 7)) * 16)) = u32x4{dw[0], dw[1], dw[2], dw[3]};
        *(u32x4*)(tb + row * 128 + (((2 * q + 1) ^ (row & 7)) * 16)) = u32x4{dw[4], dw[5], dw[6], dw[7]};
      }
#pragma unroll
      for (int it = 4 * hf; it < 4 * hf + 4; ++it) {
        const unsigned row = it * 8 + (lane >> 3);
        ov[it] = *(const u32x4*)(tb + row * 128 + (((lane & 7) ^ (row & 7)) * 16));
      }
      const unsigned chunk = (((wave >> 1) * 2 + hf) * 2 + (wave & 1)) * 128 + b;
      if ((unsigned)hf >= nearly) {
#pragma unroll
        for (int it = 4 * hf; it < 4 * hf + 4; ++it)
          st128<AUX_SYS>(ov[it], self, EXP_O + chunk * 4096 + lane * 16 + (it & 3) * 1024);
      }
    }
    wait_vm<0>();
    if (lane < 2 * NR && (lane >> 3) >= nearly && (lane & 7) != rank) {
      glb_u64* fp = (glb_u64*)(P.ws[lane & 7] + FO_SRC + (rank * 2048 + (((wave >> 1) * 2 + (lane >> 3)) * 2 + (wave & 1)) * 128 + b) * 8);
      __hip_atomic_store(fp, pair(seq, seq), __ATOMIC_RELAXED, __HIP_MEMORY_SCOPE_SYSTEM);
    }
    if (wave == 0) STAMP(4);
    __amdgpu_buffer_rsrc_t ro = mk_rsrc(P.out, OUT_BYTES);
#pragma unroll
    for (int it = 0; it < 8; ++it) st128<AUX_SC1>(ov[it], ro, osoff + ovoff + it * 8 * NDIM);
    for (unsigned job = b * 8 + wave; job < 7 * YROWS; job += 1024) {
      const unsigned p = (rank + 1 + job % 7) & 7, id = job / 7;
      const bool ok = wait_flag(self, FY_RDY + (p * 256 + id) * 8, seq);
      const float iv = ok ? inv : __uint_as_float(0x7fc00000u);
      unsigned dw[8];
#pragma unroll
      for (int r = 0; r < 4; ++r) {
        const u32x4 v = __builtin_amdgcn_raw_buffer_load_b128(self, STG_Y + (p * 256 + id) * 4096 + r * 1024 + lane * 16, 0, AUX_SYS);
#pragma unroll
        for (int hh = 0; hh < 2; ++hh) {
          int pk = __builtin_amdgcn_cvt_pk_fp8_f32(__uint_as_float(v[2 * hh] << 16) * iv, __uint_as_float(v[2 * hh] & 0xffff0000u) * iv, 0, false);
          pk = __builtin_amdgcn_cvt_pk_fp8_f32(__uint_as_float(v[2 * hh + 1] << 16) * iv, __uint_as_float(v[2 * hh + 1] & 0xffff0000u) * iv, pk, true);
          dw[2 * r + hh] = (unsigned)pk;
        }
      }
      *(u32x4*)(tb + c * 128 + (((2 * q) ^ (c & 7)) * 16)) = u32x4{dw[0], dw[1], dw[2], dw[3]};
      *(u32x4*)(tb + c * 128 + (((2 * q + 1) ^ (c & 7)) * 16)) = u32x4{dw[4], dw[5], dw[6], dw[7]};
#pragma unroll
      for (int it = 0; it < 2; ++it) {
        const unsigned row = it * 8 + (lane >> 3);
        const u32x4 o = *(const u32x4*)(tb + row * 128 + (((lane & 7) ^ (row & 7)) * 16));
        st128<AUX_SC1>(o, ro, ((id >> 4) * 16 + it * 8) * NDIM + p * NBLK + (id & 15) * 128 + ovoff);
      }
    }
  } else if (wave >= 1 && wave <= 3) {
    unsigned seq = 0;
    export_job(P, (wave - 1) * 128 + b, lane, self, seq, true);
    if (wave == 1) STAMP(1);
  } else if (wave == 0 && b < NPULLERS) {
    const unsigned seq = rfl(__builtin_amdgcn_raw_buffer_load_b32(self, EPOCH, 0, AUX_SYS)) + 1;
    const unsigned pq = b % 7, k = b / 7;
    const unsigned p = (rank + 1 + pq) & 7;
    const unsigned rotx = 8 * ((pq * 15 + 3) / 7);
    const unsigned voff = lane * 16;
    const u32x4 dp = mk_desc(P.ws[p], WS_NEED), dout = mk_desc(P.out, OUT_BYTES);
    const unsigned ovoff = (lane >> 3) * NDIM + (lane & 7) * 16;
    const unsigned ovo[4] = {ovoff, ovoff + 8 * NDIM, ovoff + 16 * NDIM, ovoff + 24 * NDIM};
    const u32x2 seq2 = {seq, seq};
    const u32x4 nan4 = {0x7f7f7f7fu, 0x7f7f7f7fu, 0x7f7f7f7fu, 0x7f7f7f7fu};
    auto run = [&](auto phase_tag) {
      constexpr int PH = decltype(phase_tag)::value;
      constexpr unsigned J0 = PH == 0 ? 4 * PUSH_W : 0;
      constexpr unsigned NJ = PH == 0 ? 1536u + YROWS : 2048u - YROWS / 2;
      constexpr unsigned NP = PH == 0 ? NP_IN : NP_OUT;
      if (k >= NP) return;
      const unsigned tk_off = TICKET + (PH * 8 + pq) * 128;
      const unsigned tk_base = (seq - 1) * (NJ - J0 - 2 * NP) - 3 * NP - J0;
      auto decode = [&](unsigned n, unsigned& kind, unsigned& idx) {
        if (n < 72) { kind = n < 64; idx = n < 64 ? p * 512 + n : p * 1024 + n - 64; return; }
        if (n >= 1536) { kind = 2; idx = n - 1536; return; }
        const unsigned s = (n - 72) / 192, r = (n - 72) % 192, t = r / 3, e = r % 3;
        if (s < 7 && e == 2) { kind = 1; idx = (p * 8 + s + 1) * 64 + ((t + 8 * (pq + 1)) & 63); return; }
        kind = 0;
        const unsigned xi = s < 7 ? t * 2 + e : r + 8;
        if (xi < 8) { idx = (p * 8 + s + 1) * 128 + xi; return; }
        unsigned cx = xi - 8 + rotx;
        if (cx >= 120) cx -= 120;
        idx = (p * 8 + s) * 128 + 8 + cx;
      };
      auto ochunk = [&](unsigned n) -> unsigned {
        constexpr unsigned G0 = YROWS / 16, FULL = (16 - G0) * 128;
        if (n < FULL) return G0 * 128 + n;
        const unsigned g = (n - FULL) / 120, i = (n - FULL) % 120;
        return g * 128 + (i < 24 ? (i / 6) * 8 + 2 + i % 6 : i + 8);
      };
      auto f_src = [&](unsigned n) -> unsigned {
        if constexpr (PH == 1) return FO_SRC + (p * 2048 + ochunk(n)) * 8;
        unsigned kind, idx;
        decode(n, kind, idx);
        return kind == 0 ? FX_SRC + idx * 8 : kind == 1 ? FW_SRC + (idx >> 2) * 8 : FY_SRC + (p * 256 + idx) * 8;
      };
      auto addrs = [&](unsigned n, unsigned& src, unsigned& dst, unsigned& frdy) {
        if constexpr (PH == 1) {
          const unsigned ch = ochunk(n);
          const unsigned tb = ch & 127, g = tb & 7, x = tb >> 3;
          const unsigned i = (g >> 1) * 4 + (x >> 2), j = (g & 1) * 4 + (x & 3);
          src = EXP_O + ch * 4096;
          dst = (i * 256 + (ch >> 8) * 32) * NDIM + p * NBLK + j * 256 + ((ch >> 7) & 1) * 128;
          frdy = 0;
        } else {
          unsigned kind, idx;
          decode(n, kind, idx);
          if (kind == 2) {
            src = EXP_Y + idx * 4096;
            dst = STG_Y + (p * 256 + idx) * 4096;
            frdy = FY_RDY + (p * 256 + idx) * 8;
          } else {
            dst = (kind ? STG_W : STG_X) + idx * 4096;
            src = kind ? EXP_W + (rank * 512 + (idx & 511)) * 4096 : dst;
            frdy = (kind ? FW_RDY : FX_RDY) + idx * 8;
          }
        }
      };
      bool seen7 = false; (void)seen7;
      u32x4 va[4], vb[4];
      unsigned j1 = J0 + k + NP, j2 = J0 + k + 2 * NP;
      unsigned src, dst, frdy, pfrdy;
      bool ok = wait_flag(self, f_src(J0 + k), seq);
      STAMP(PH == 0 ? 33 : 35);
      addrs(J0 + k, src, dst, frdy);
      u32x2 fv;
      unsigned dummy = 0;
      a_ld4(va, voff, dp, src);
      a_ld2(fv, ds, f_src(j1));
      unsigned nsrc, ndst, nfrdy;
      addrs(j1, nsrc, ndst, nfrdy);
      a_wait<0>(va, dummy, fv);
      auto step = [&](u32x4 (&cur)[4], u32x4 (&nxt)[4]) -> bool {
        if (j1 >= NJ) return false;
        const bool cur_ok = ok;
        if (rfl(min(fv[0], fv[1])) < seq) ok = wait_flag(self, f_src(j1), seq) && ok;
        a_ld4(nxt, voff, dp, nsrc);
        unsigned tk = 1;
        const bool has2 = j2 < NJ;
        if (has2) a_ticket_flag(tk, fv, ds, tk_off, f_src(j2));
        const unsigned cdst = dst;
        pfrdy = frdy; dst = ndst; frdy = nfrdy;
        if constexpr (PH == 0) {
          a_st4_sc1(cur, voff, ds, cdst);
        } else {
          if (!cur_ok) { cur[0] = nan4; cur[1] = nan4; cur[2] = nan4; cur[3] = nan4; }
          a_st4_rows(cur, ovo, dout, cdst);
        }
        j1 = j2;
        addrs(has2 ? j1 : J0, nsrc, ndst, nfrdy);
        asm volatile("" : "+s"(nsrc), "+s"(ndst), "+s"(nfrdy));
        if constexpr (PH == 0) {
          a_wait<0>(nxt, tk, fv);
          a_st2_sc1(seq2, ds, pfrdy);
        } else {
          a_wait<4>(nxt, tk, fv);
        }

        j2 = has2 ? rfl(tk) - tk_base : NJ;
        if (j2 > NJ) j2 = NJ;
        return true;
      };
      auto last = [&](u32x4 (&cur)[4]) {
        if constexpr (PH == 0) {
          a_st4_sc1(cur, voff, ds, dst);
          wait_vm<0>();
          a_st2_sc1(seq2, ds, frdy);
        } else {
          if (!ok) { cur[0] = nan4; cur[1] = nan4; cur[2] = nan4; cur[3] = nan4; }
          a_st4_rows(cur, ovo, dout, dst);
        }
      };
      for (;;) {
        if (!step(va, vb)) { last(va); break; }
        if (!step(vb, va)) { last(vb); break; }
      }
    };
    STAMP(32);
    run(std::integral_constant<int, 0>{});
    STAMP(34);
    run(std::integral_constant<int, 1>{});
    STAMP(36);
  } else if (wave == 0 && b == REDUCER) {
    const unsigned seq = rfl(__builtin_amdgcn_raw_buffer_load_b32(self, EPOCH, 0, AUX_SYS)) + 1;
    unsigned gm = 0;
    unsigned long long t0 = now();
    for (unsigned spins = 0;; ++spins) {
      asm volatile("" ::: "memory");
      u32x4 a = __builtin_amdgcn_raw_buffer_load_b128(self, AMAX_WG + lane * 16, 0, AUX_SYS);
      gm = max(a[0], a[2]);
      if (__all(a[1] >= seq && a[3] >= seq)) break;
      if ((spins & 63) == 63 && give_up(self, seq, t0)) break;
    }
    gm = wave_max_u32(gm);
    glb_u64* gp = (glb_u64*)(P.ws[lane >> 3] + AMAX_G + (lane & 7) * 128 + rank * 8);
    __hip_atomic_store(gp, pair(gm, seq), __ATOMIC_RELAXED, __HIP_MEMORY_SCOPE_SYSTEM);
    if (lane == 0) __builtin_amdgcn_raw_buffer_store_b32(seq, self, EPOCH, 0, AUX_SYS);
    STAMP(39);
  }
}

}

extern "C" void kernel_launch(int rank, int nranks, ncclComm_t comm, hipStream_t stream, void* const* d_in,
                              const int* in_sizes, int n_in, void* d_out, int out_size, const int* in_dtypes,
                              int out_dtype, void* const* d_out_all, void* const* d_ws_all, size_t ws_bytes) {
  (void)comm; (void)in_sizes; (void)in_dtypes; (void)out_dtype; (void)out_size; (void)d_out_all;
  if (nranks != NR || n_in < 2 || (unsigned)rank >= (unsigned)NR || ws_bytes < WS_NEED) {
    fprintf(stderr, "kernel_launch: unsupported configuration\n");
    return;
  }
  Params P;
  P.x = (const char*)d_in[0];
  P.w = (const char*)d_in[1];
  for (int p = 0; p < NR; ++p) P.ws[p] = (char*)d_ws_all[p];
  P.out = (char*)d_out;
  P.rank = rank;
  hipLaunchKernelGGL(fused_kernel, dim3(NWG), dim3(NTHREADS), 0, stream, P);
  hipError_t e = hipGetLastError();
  if (e != hipSuccess) fprintf(stderr, "launch failed: %s\n", hipGetErrorString(e));
}
```

```cpp
#include <hip/hip_runtime.h>
#include <rccl/rccl.h>
#include <cstdio>
#include <cstdint>
#include <type_traits>

namespace {

constexpr int NR = 8;
constexpr int MDIM = 4096, KSH = 512, NDIM = 16384, NBLK = 2048;
constexpr int NWG = 256;
constexpr int NTHREADS = 512;
constexpr unsigned NP = 16;
constexpr unsigned NPULLERS = 7 * NP;
constexpr unsigned REDUCER = 127;
constexpr unsigned PUSH_W = 8;
constexpr int PACE_W = 3;
constexpr unsigned YROWS = 160;
constexpr int POLL_SLEEP = 4, EXPORT_DELAY = 64, EXPORT_GAP = 32, OUT_DELAY = 8;
static_assert(NPULLERS <= REDUCER && 14 * PUSH_W <= 192 && YROWS % 32 == 0 && YROWS < 256, "role layout");
constexpr unsigned long long DEADLINE_TICKS = 100000000ull;

constexpr unsigned STG_X = 0;
constexpr unsigned STG_W = 32u << 20;
constexpr unsigned EXP_W = 48u << 20;
constexpr unsigned EXP_O = 64u << 20;
constexpr unsigned FLG = 72u << 20;
constexpr unsigned FX_SRC = FLG;
constexpr unsigned FX_RDY = FLG + (64u << 10);
constexpr unsigned FW_SRC = FLG + (128u << 10);
constexpr unsigned FW_RDY = FLG + (144u << 10);
constexpr unsigned FO_SRC = FLG + (192u << 10);
constexpr unsigned AMAX_WG = FLG + (320u << 10);
constexpr unsigned AMAX_G = FLG + (324u << 10);
constexpr unsigned EPOCH = FLG + (328u << 10);
constexpr unsigned TMO = EPOCH + 256;
constexpr unsigned TICKET = FLG + (332u << 10);
constexpr unsigned FY_SRC = FLG + (336u << 10);
constexpr unsigned FY_RDY = FLG + (352u << 10);
constexpr unsigned EXP_Y = FLG + (1u << 20);
constexpr unsigned STG_Y = FLG + (2u << 20);
constexpr unsigned WS_NEED = FLG + (10u << 20);

constexpr int SLOT_BYTES = 32 * 1024;
constexpr int L_CTRL = 4 * SLOT_BYTES;
constexpr int L_AMAX = L_CTRL;
constexpr int L_ACNT = L_CTRL + 4;
constexpr int L_SCALE = L_CTRL + 8;
constexpr int L_SFLAG = L_CTRL + 12;
constexpr int L_FLG = L_CTRL + 64;
constexpr int LDS_BYTES = L_FLG + 2048;

typedef __attribute__((ext_vector_type(8))) short bf16x8;
typedef __attribute__((ext_vector_type(4))) float f32x4;
typedef __attribute__((ext_vector_type(4))) unsigned u32x4;
typedef __attribute__((ext_vector_type(2))) unsigned u32x2;
typedef __attribute__((ext_vector_type(2))) float f32x2;
typedef __attribute__((ext_vector_type(2))) __bf16 bf16x2;
typedef __attribute__((address_space(3))) unsigned lds_u32;
typedef __attribute__((address_space(1))) unsigned glb_u32;
typedef __attribute__((address_space(1))) unsigned long long glb_u64;

struct Params {
  const char* x;
  const char* w;
  char* out;
  char* ws[NR];
  int rank;
};

constexpr int AUX_SC1 = 16, AUX_SYS = 17;
constexpr unsigned OUT_BYTES = (unsigned)MDIM * NDIM;

__device__ __forceinline__ __amdgpu_buffer_rsrc_t mk_rsrc(const void* p, unsigned bytes) {
  return __builtin_amdgcn_make_buffer_rsrc(const_cast<void*>(p), 0, bytes, 0x00020000);
}
template <int AUX> __device__ __forceinline__ void st128(u32x4 v, __amdgpu_buffer_rsrc_t r, unsigned off) {
  __builtin_amdgcn_raw_buffer_store_b128(v, r, off, 0, AUX);
}
template <int N> __device__ __forceinline__ void wait_vm() { asm volatile("s_waitcnt vmcnt(%0)" ::"i"(N) : "memory"); }
__device__ __forceinline__ void wait_lds() {
  asm volatile("" ::: "memory");
  __builtin_amdgcn_s_waitcnt(0xC07F);
  asm volatile("" ::: "memory");
}
__device__ __forceinline__ unsigned rfl(unsigned v) { return __builtin_amdgcn_readfirstlane(v); }
__device__ __forceinline__ unsigned lds_ld(const char* smem, int off) {
  return rfl(__hip_atomic_load((const lds_u32*)(smem + off), __ATOMIC_RELAXED, __HIP_MEMORY_SCOPE_WORKGROUP));
}
__device__ __forceinline__ void lds_st(char* smem, int off, unsigned v) {
  __hip_atomic_store((lds_u32*)(smem + off), v, __ATOMIC_RELAXED, __HIP_MEMORY_SCOPE_WORKGROUP);
}
__device__ __forceinline__ unsigned wave_max_u32(unsigned v) {
  v = max(v, (unsigned)__builtin_amdgcn_update_dpp(0, (int)v, 0x111, 0xf, 0xf, false));
  v = max(v, (unsigned)__builtin_amdgcn_update_dpp(0, (int)v, 0x112, 0xf, 0xf, false));
  v = max(v, (unsigned)__builtin_amdgcn_update_dpp(0, (int)v, 0x114, 0xf, 0xf, false));
  v = max(v, (unsigned)__builtin_amdgcn_update_dpp(0, (int)v, 0x118, 0xf, 0xf, false));
  const unsigned a = __builtin_amdgcn_readlane(v, 15), b = __builtin_amdgcn_readlane(v, 31);
  const unsigned c = __builtin_amdgcn_readlane(v, 47), d = __builtin_amdgcn_readlane(v, 63);
  return max(max(a, b), max(c, d));
}
__device__ __forceinline__ unsigned long long now() { return __builtin_amdgcn_s_memrealtime(); }
__device__ __forceinline__ unsigned long long pair(unsigned lo, unsigned hi) { return ((unsigned long long)hi << 32) | lo; }


__device__ __forceinline__ bool give_up(__amdgpu_buffer_rsrc_t self, unsigned seq, unsigned long long t0) {
  unsigned t = rfl(__builtin_amdgcn_raw_buffer_load_b32(self, TMO, 0, AUX_SYS));
  if (t == seq || now() - t0 > DEADLINE_TICKS) {
    __builtin_amdgcn_raw_buffer_store_b32(seq, self, TMO, 0, AUX_SYS);
    return true;
  }
  return false;
}

__device__ __forceinline__ bool wait_flag(__amdgpu_buffer_rsrc_t self, unsigned off, unsigned seq) {
  unsigned long long t0 = now();
  for (unsigned spins = 0;; ++spins) {
    asm volatile("" ::: "memory");
    u32x2 v = __builtin_amdgcn_raw_buffer_load_b64(self, off, 0, AUX_SYS);
    if (__all(v[0] >= seq && v[1] >= seq)) return true;
    if ((spins & 63) == 63 && give_up(self, seq, t0)) return false;
    __builtin_amdgcn_s_sleep(1);
  }
}

__device__ __forceinline__ u32x4 mk_desc(const void* p, unsigned bytes) {
  const unsigned long long a = (unsigned long long)p;
  return u32x4{(unsigned)a, (unsigned)(a >> 32) & 0xffffu, bytes, 0x00020000u};
}
template <int H>
__device__ __forceinline__ void glds22(unsigned voff, const char* wsrc, const char* xsrc, unsigned lw, unsigned lx) {
  unsigned keep;
  asm volatile(
      "s_mov_b32 %0, m0\n\ts_mov_b32 m0, %4\n\ts_nop 0\n\t"
      "global_load_lds_dwordx4 %1, %2 offset:%6 sc1\n\t"
      "global_load_lds_dwordx4 %1, %2 offset:%7 sc1\n\t"
      "s_mov_b32 m0, %5\n\ts_nop 0\n\t"
      "global_load_lds_dwordx4 %1, %3 offset:%6 sc1\n\t"
      "global_load_lds_dwordx4 %1, %3 offset:%7 sc1\n\t"
      "s_mov_b32 m0, %0"
      : "=&s"(keep) : "v"(voff), "s"(wsrc), "s"(xsrc), "s"(lw), "s"(lx), "i"(H * 2048), "i"(H * 2048 + 1024) : "memory");
}
__device__ __forceinline__ void a_ld4(u32x4 (&d)[4], unsigned voff, u32x4 rs, unsigned soff) {
  asm volatile(
      "buffer_load_dwordx4 %0, %4, %5, %6 offen sc0 sc1\n\t"
      "buffer_load_dwordx4 %1, %4, %5, %6 offen offset:1024 sc0 sc1\n\t"
      "buffer_load_dwordx4 %2, %4, %5, %6 offen offset:2048 sc0 sc1\n\t"
      "buffer_load_dwordx4 %3, %4, %5, %6 offen offset:3072 sc0 sc1"
      : "=&v"(d[0]), "=&v"(d[1]), "=&v"(d[2]), "=&v"(d[3]) : "v"(voff), "s"(rs), "s"(soff) : "memory");
}
__device__ __forceinline__ void a_st4_sc1(const u32x4 (&d)[4], unsigned voff, u32x4 rs, unsigned soff) {
  asm volatile(
      "buffer_store_dwordx4 %0, %4, %5, %6 offen sc1\n\t"
      "buffer_store_dwordx4 %1, %4, %5, %6 offen offset:1024 sc1\n\t"
      "buffer_store_dwordx4 %2, %4, %5, %6 offen offset:2048 sc1\n\t"
      "buffer_store_dwordx4 %3, %4, %5, %6 offen offset:3072 sc1\n\ts_nop 1"
      :: "v"(d[0]), "v"(d[1]), "v"(d[2]), "v"(d[3]), "v"(voff), "s"(rs), "s"(soff) : "memory");
}
__device__ __forceinline__ void a_st4_rows(const u32x4 (&d)[4], const unsigned (&vo)[4], u32x4 rs, unsigned soff) {
  asm volatile(
      "buffer_store_dwordx4 %0, %4, %8, %9 offen sc1\n\t"
      "buffer_store_dwordx4 %1, %5, %8, %9 offen sc1\n\t"
      "buffer_store_dwordx4 %2, %6, %8, %9 offen sc1\n\t"
      "buffer_store_dwordx4 %3, %7, %8, %9 offen sc1\n\ts_nop 1"
      :: "v"(d[0]), "v"(d[1]), "v"(d[2]), "v"(d[3]), "v"(vo[0]), "v"(vo[1]), "v"(vo[2]), "v"(vo[3]), "s"(rs), "s"(soff) : "memory");
}
__device__ __forceinline__ void a_ld2(u32x2& d, u32x4 rs, unsigned soff) {
  asm volatile("buffer_load_dwordx2 %0, off, %1, %2 sc0 sc1" : "=&v"(d) : "s"(rs), "s"(soff) : "memory");
}
__device__ __forceinline__ void glds_flag(unsigned voff, const char* base, unsigned l) {
  unsigned keep;
  asm volatile(
      "s_mov_b32 %0, m0\n\ts_mov_b32 m0, %3\n\ts_nop 0\n\t"
      "global_load_lds_dwordx4 %1, %2 sc0 sc1\n\t"
      "s_mov_b32 m0, %0"
      : "=&s"(keep) : "v"(voff), "s"(base), "s"(l) : "memory");
}
__device__ __forceinline__ void a_st2_sc1(u32x2 v, u32x4 rs, unsigned soff) {
  asm volatile("buffer_store_dwordx2 %0, off, %1, %2 sc1" :: "v"(v), "s"(rs), "s"(soff) : "memory");
}
__device__ __forceinline__ void a_ticket_flag(unsigned& tk, u32x2& fv, u32x4 rs, unsigned tk_off, unsigned f_off) {
  asm volatile(
      "s_mov_b64 exec, 1\n\t"
      "buffer_atomic_add %0, off, %2, %3 sc0\n\t"
      "s_mov_b64 exec, -1\n\t"
      "buffer_load_dwordx2 %1, off, %2, %4 sc0 sc1"
      : "+v"(tk), "+v"(fv) : "s"(rs), "s"(tk_off), "s"(f_off) : "memory");
}
template <int N> __device__ __forceinline__ void a_wait(u32x4 (&d)[4], unsigned& a, u32x2& b) {
  asm volatile("s_waitcnt vmcnt(%6)" : "+v"(d[0]), "+v"(d[1]), "+v"(d[2]), "+v"(d[3]), "+v"(a), "+v"(b) : "i"(N) : "memory");
}

__device__ __forceinline__ void export_job(const Params& P, unsigned n, unsigned lane, __amdgpu_buffer_rsrc_t self,
                                           unsigned& seq, bool first) {
  const unsigned rank = P.rank;
  const unsigned s = n / 384, r = n % 384;
  const unsigned sg = rank * 8 + s;
  if (r < 256) {
    const unsigned d = r & 7, j = r >> 4, h = (r >> 3) & 1;
    __amdgpu_buffer_rsrc_t rw = mk_rsrc(P.w, KSH * NDIM * 2);
    const unsigned o = lane & 15, g = lane >> 4;
    const unsigned voff = ((g * 8) * NDIM + o * 8) * 2;
    const unsigned soff = ((s * 64 + h * 32) * NDIM + d * NBLK + j * 128) * 2;
    u32x4 row[8];
#pragma unroll
    for (int jj = 0; jj < 8; ++jj) row[jj] = __builtin_amdgcn_raw_buffer_load_b128(rw, voff + jj * NDIM * 2, soff, 0);
    if (first) seq = rfl(__builtin_amdgcn_raw_buffer_load_b32(self, EPOCH, 0, AUX_SYS)) + 1;
    const bool direct = d == rank || (s == 0 && j < PUSH_W);
    char* const base = P.ws[direct ? d : rank];
    __amdgpu_buffer_rsrc_t rd = mk_rsrc(base, WS_NEED);
    const unsigned dst = (direct ? STG_W + sg * 16 * 16384 : EXP_W + (d * 8 + s) * 16 * 16384) + j * 16384 + h * 2048;
#pragma unroll
    for (int t = 0; t < 8; ++t) {
      u32x4 f;
#pragma unroll
      for (int e = 0; e < 4; ++e)
        f[e] = __builtin_amdgcn_perm(row[2 * e + 1][t >> 1], row[2 * e][t >> 1], (t & 1) ? 0x07060302u : 0x05040100u);
      st128<AUX_SYS>(f, rd, dst + lane * 16 + (t & 1) * 1024 + (t >> 1) * 4096);
      if (direct && d != rank) wait_vm<PACE_W - 1>();
    }
    wait_vm<0>();
    if (direct) {
      if (lane < 4) __builtin_amdgcn_raw_buffer_store_b32(seq, rd, FW_RDY + ((sg * 16 + j) * 4 + lane) * 8 + h * 4, 0, AUX_SYS);
    } else if (lane == 0) {
      __amdgpu_buffer_rsrc_t rp = mk_rsrc(P.ws[d], WS_NEED);
      __builtin_amdgcn_raw_buffer_store_b32(seq, rp, FW_SRC + (sg * 16 + j) * 8 + h * 4, 0, AUX_SYS);
    }
  } else {
    const unsigned i = (r - 256) >> 3, sub = (r - 256) & 7;
    __amdgpu_buffer_rsrc_t rx = mk_rsrc(P.x, MDIM * KSH * 2);
    const unsigned c = lane & 15, q = lane >> 4;
    const unsigned voff = (c * KSH + q * 8) * 2;
    const unsigned soff = ((i * 256 + sub * 32) * KSH + s * 64) * 2;
    u32x4 f[4];
#pragma unroll
    for (int b = 0; b < 4; ++b)
      f[b] = __builtin_amdgcn_raw_buffer_load_b128(rx, voff + ((b & 1) * 16 * KSH + (b >> 1) * 32) * 2, soff, 0);
    if (first) seq = rfl(__builtin_amdgcn_raw_buffer_load_b32(self, EPOCH, 0, AUX_SYS)) + 1;
    const unsigned idx = (sg * 16 + i) * 8 + sub;
#pragma unroll
    for (int b = 0; b < 4; ++b) st128<AUX_SYS>(f[b], self, STG_X + idx * 4096 + lane * 16 + b * 1024);
    wait_vm<0>();
    if (lane < NR) {
      glb_u64* fp = (glb_u64*)(P.ws[lane] + (lane == rank ? FX_RDY : FX_SRC) + idx * 8);
      __hip_atomic_store(fp, pair(seq, seq), __ATOMIC_RELAXED, __HIP_MEMORY_SCOPE_SYSTEM);
    }
  }
}

__global__ void __launch_bounds__(NTHREADS, 2) fused_kernel(Params P) {
  __shared__ __attribute__((aligned(16))) unsigned smem32[LDS_BYTES / 4];
  char* const smem = (char*)smem32;
  const unsigned lane = threadIdx.x & 63;
  const unsigned wave = rfl(threadIdx.x >> 6);
  const unsigned rank = P.rank;
  char* const wsl = P.ws[rank];
  __amdgpu_buffer_rsrc_t self = mk_rsrc(wsl, WS_NEED);
  const u32x4 ds = mk_desc(wsl, WS_NEED);
  const bool is_gemm = blockIdx.x >= 128;
  const unsigned b = blockIdx.x & 127;

  if (is_gemm) {
    const unsigned xg = b & 7, xi = b >> 3;
    const unsigned ti = (xg >> 1) * 4 + (xi >> 2), tj = (xg & 1) * 4 + (xi & 3);
    if (threadIdx.x < 16) smem32[L_CTRL / 4 + threadIdx.x] = 0;
    unsigned seq = 0;
    {
      constexpr unsigned NA = 14 * (16 - PUSH_W), NE = NA + 2848;
      auto elist = [&](unsigned e) -> unsigned {
        if (e < NA) return (PUSH_W + (e / 14)) * 16 + ((e / 7) & 1) * 8 + ((rank + 1 + e % 7) & 7);
        e -= NA;
        if (e < 128) return 256 + e;
        if (e < 136) return 384 + 256 + e - 128;
        if (e < 392) return 384 + e - 136;
        if (e < 512) return 384 + 264 + e - 392;
        if (e < 544) return (e - 512) * 8 + rank;
        return 768 + e - 544;
      };
      if (b >= 64) {
        export_job(P, elist(wave * 64 + b - 64), lane, self, seq, true);
      } else if (wave < 3) {
        const unsigned t = wave * 64 + b;
        if (t < 14 * PUSH_W) export_job(P, (t / 14) * 16 + ((t / 7) & 1) * 8 + ((rank + 1 + t % 7) & 7), lane, self, seq, true);
        else seq = rfl(__builtin_amdgcn_raw_buffer_load_b32(self, EPOCH, 0, AUX_SYS)) + 1;
      } else {
        __builtin_amdgcn_s_sleep(EXPORT_DELAY);
        export_job(P, elist(512 + (wave - 3) * 64 + b), lane, self, seq, true);
      }
      for (unsigned e = 832 + wave * 128 + b; e < NE; e += 1024) {
        for (unsigned i = 0; i <= wave; ++i) __builtin_amdgcn_s_sleep(EXPORT_GAP);
        export_job(P, elist(e), lane, self, seq, false);
      }
    }

    f32x4 acc[8][4];
#pragma unroll
    for (int t = 0; t < 8; ++t)
#pragma unroll
      for (int m = 0; m < 4; ++m) acc[t][m] = f32x4{0.f, 0.f, 0.f, 0.f};
    const unsigned lds0 = (unsigned)(uintptr_t)(lds_u32*)smem32 + wave * 2048;
    const unsigned voff = lane * 16;
    const char* xsrc = wsl + STG_X + (ti * 8 + wave) * 4096;
    const char* wsrc = wsl + STG_W + (tj * 8 + wave) * 4096;
    unsigned sp = 0;
    const unsigned fbase = (lane & 4) ? FW_RDY + ((lane >> 3) * 512 + tj * 8) * 8 + (lane & 3) * 16
                                      : FX_RDY + ((lane >> 3) * 1024 + ti * 8) * 8 + (lane & 3) * 16;
    const unsigned fsh = (lane & 4) ? 9 : 10;
    const unsigned lpad = (unsigned)(uintptr_t)(lds_u32*)smem32 + L_FLG;
    const unsigned lpadv = lpad + lane * 16;
    unsigned tick = 0;
    bool dead = false;
    auto request = [&]() {
      if (!dead) {
        glds_flag(fbase + ((sp & 7) << fsh), wsl, lpad + (tick & 1) * 1024);
      } else {
        const u32x4 all = {seq, seq, seq, seq};
        asm volatile("ds_write_b128 %0, %1\n\ts_waitcnt lgkmcnt(0)" :: "v"(lpadv + (tick & 1) * 1024), "v"(all) : "memory");
      }
    };
    unsigned issued = 0;
    const char* wcur = wsrc;
    const char* xcur = xsrc;
    const char* const padp = smem + L_FLG + lane * 16;
    auto snap = [&]() -> u32x4 { return *(const u32x4*)(padp + ((tick + 1) & 1) * 1024); };
    auto pick = [&](const u32x4 f) -> bool {
      unsigned long long m = __ballot(min(min(f[0], f[1]), min(f[2], f[3])) >= seq && sp < 8);
      m &= m >> 4; m &= m >> 2; m &= m >> 1; m &= 0x0101010101010101ull;
      if (m == 0) return false;
      const unsigned l8 = __builtin_ctzll(m);
      const unsigned sg = l8 + __builtin_amdgcn_readlane(sp, l8);
      sp += (lane >> 3) == (l8 >> 3) ? 1 : 0;
      wcur = wsrc + (size_t)sg * (64 * 4096);
      xcur = xsrc + (size_t)sg * (128 * 4096);
      return true;
    };
    auto issue_even = [&]() {
      const unsigned l = lds0 + (issued & 2) * SLOT_BYTES;
      glds22<0>(voff, wcur, xcur, l, l + 16384);
      ++issued;
    };
    auto issue_odd = [&]() {
      const unsigned l = lds0 + (issued & 2) * SLOT_BYTES + SLOT_BYTES - 2048;
      glds22<1>(voff, wcur, xcur, l, l + 16384);
      ++issued;
    };
    const char* wb = smem + (wave & 1) * 8192 + lane * 16;
    const char* xb = smem + 16384 + (wave >> 1) * 4096 + lane * 16;
    bf16x8 wa[8], xa[4], wc[8], xc[4];
    auto mmh = [&](bf16x8 (&wf)[8], bf16x8 (&xf)[4], auto h_tag) {
      constexpr int H = decltype(h_tag)::value;
      __builtin_amdgcn_sched_barrier(0);
#pragma unroll
      for (int t = 4 * H; t < 4 * H + 4; ++t)
#pragma unroll
        for (int m = 0; m < 4; ++m) acc[t][m] = __builtin_amdgcn_mfma_f32_16x16x32_bf16(wf[t], xf[m], acc[t][m], 0, 0, 0);
      __builtin_amdgcn_sched_barrier(0);
    };
    constexpr std::integral_constant<int, 0> H0{};
    constexpr std::integral_constant<int, 1> H1{};
    auto mm = [&](bf16x8 (&wf)[8], bf16x8 (&xf)[4]) { mmh(wf, xf, H0); mmh(wf, xf, H1); };
    auto rd = [&](bf16x8 (&wf)[8], bf16x8 (&xf)[4], unsigned so) {
#pragma unroll
      for (int m = 0; m < 4; ++m) xf[m] = *(const bf16x8*)(xb + so + m * 1024);
#pragma unroll
      for (int t = 0; t < 8; ++t) wf[t] = *(const bf16x8*)(wb + so + t * 1024);
    };
    unsigned hs = 0, young = 0;
    auto top = [&]() {
      const unsigned ahead = issued - hs;
      unsigned allow = ahead > 1 ? ahead - 1 : 0;
      if (wave == 0 && young < allow && issued < 127) allow = young;
      if (allow == 0) wait_vm<0>(); else if (allow == 1) wait_vm<4>(); else wait_vm<8>();
      __builtin_amdgcn_s_barrier();
      asm volatile("" ::: "memory");
    };
    auto stage = [&](const u32x4 f) {
      const unsigned lim = hs + 3 < 127 ? hs + 3 : 127, before = issued;
      const char* wodd = wcur;
      const char* xodd = xcur;
      bool got = false;
      if (issued + (issued & 1) <= lim && tick) got = pick(f);
      if (wave == 0 && issued + (got ? 2 : 0) < 127) request();
      if ((issued & 1) && issued <= lim) {
        const unsigned l = lds0 + (issued & 2) * SLOT_BYTES + SLOT_BYTES - 2048;
        glds22<1>(voff, wodd, xodd, l, l + 16384);
        ++issued;
      }
      if (got) {
        issue_even();
        if (issued <= lim) issue_odd();
      }
      young = issued - before;
      ++tick;
    };
    auto rd_mm = [&](bf16x8 (&wf)[8], bf16x8 (&xf)[4], bf16x8 (&wp)[8], bf16x8 (&xp)[4]) {
      __builtin_amdgcn_sched_barrier(0);
      rd(wf, xf, (hs & 3) * SLOT_BYTES);
#pragma unroll
      for (int t = 0; t < 4; ++t)
#pragma unroll
        for (int m = 0; m < 4; ++m) acc[t][m] = __builtin_amdgcn_mfma_f32_16x16x32_bf16(wp[t], xp[m], acc[t][m], 0, 0, 0);
#pragma unroll
      for (int g = 0; g < 4; ++g) {
        __builtin_amdgcn_sched_group_barrier(0x008, 1, 0);
        __builtin_amdgcn_sched_group_barrier(0x100, 1, 0);
        __builtin_amdgcn_sched_group_barrier(0x008, 1, 0);
        __builtin_amdgcn_sched_group_barrier(0x100, 1, 0);
        __builtin_amdgcn_sched_group_barrier(0x008, 1, 0);
        __builtin_amdgcn_sched_group_barrier(0x100, 1, 0);
        __builtin_amdgcn_sched_group_barrier(0x008, 1, 0);
      }
      __builtin_amdgcn_sched_barrier(0);
    };
    bool deferred = false;
    while (hs < 128) {
      while (issued == hs + 3 && deferred && young == 1 && hs < 124) {
        wait_vm<8>();
        __builtin_amdgcn_s_barrier();
        asm volatile("" ::: "memory");
        rd_mm(wa, xa, wc, xc);
        issue_odd();
        mmh(wc, xc, H1);
        ++hs;
        wait_lds();
        wait_vm<8>();
        __builtin_amdgcn_s_barrier();
        asm volatile("" ::: "memory");
        const u32x4 f = snap();
        rd_mm(wc, xc, wa, xa);
        const bool got = pick(f);
        if (wave == 0) request();
        ++tick;
        if (got) issue_even();
        young = got ? 1 : 0;
        mmh(wa, xa, H1);
        ++hs;
        wait_lds();
      }
      if (issued == hs) {
        if (deferred) mm(wc, xc);
        deferred = false;
        const unsigned long long t0 = now();
        for (unsigned idle = 1;; ++idle) {
          top();
          stage(snap());
          if (issued != hs) break;
          if (wave == 0 && (idle & 63) == 0 && give_up(self, seq, t0)) dead = true;
          __builtin_amdgcn_s_sleep(POLL_SLEEP);
        }
      }
      top();
      {
        const u32x4 f = snap();
        if (deferred) rd_mm(wa, xa, wc, xc); else rd(wa, xa, (hs & 3) * SLOT_BYTES);
        stage(f);
      }
      if (deferred) mmh(wc, xc, H1);
      wait_lds();
      __builtin_amdgcn_sched_barrier(0);
      ++hs;
      top();
      {
        const u32x4 f = snap();
        rd_mm(wc, xc, wa, xa);
        stage(f);
      }
      mmh(wa, xa, H1);
      wait_lds();
      __builtin_amdgcn_sched_barrier(0);
      ++hs;
      deferred = true;
    }
    mm(wc, xc);
    float mx = 0.f;
#pragma unroll
    for (int t = 0; t < 8; ++t)
#pragma unroll
      for (int m = 0; m < 4; ++m)
#pragma unroll
        for (int r = 0; r < 4; ++r) mx = fmaxf(mx, fabsf(acc[t][m][r]));
    const unsigned wmx = wave_max_u32(__float_as_uint(mx));
    if (lane == 0) {
      __hip_atomic_fetch_max((lds_u32*)(smem + L_AMAX), wmx, __ATOMIC_RELAXED, __HIP_MEMORY_SCOPE_WORKGROUP);
      asm volatile("" ::: "memory");
      const unsigned cnt = __hip_atomic_fetch_add((lds_u32*)(smem + L_ACNT), 1u, __ATOMIC_RELAXED, __HIP_MEMORY_SCOPE_WORKGROUP);
      asm volatile("" ::: "memory");
      const unsigned am = __hip_atomic_load((lds_u32*)(smem + L_AMAX), __ATOMIC_RELAXED, __HIP_MEMORY_SCOPE_WORKGROUP);
      asm volatile("" ::: "memory");
      if (cnt == 7) {
        __builtin_amdgcn_raw_buffer_store_b64(u32x2{am, seq}, self, AMAX_WG + b * 8, 0, AUX_SYS);
      }
    }
    const unsigned nearly = ti != 0 || YROWS / 32 <= (wave >> 1) * 2 ? 0 : min(2u, YROWS / 32 - (wave >> 1) * 2);
    if (nearly) {
#pragma unroll
      for (int m = 0; m < 4; ++m) {
        if ((unsigned)(m >> 1) >= nearly) break;
        const unsigned yo = EXP_Y + (((wave >> 1) * 4 + m) * 16 + tj * 2 + (wave & 1)) * 4096 + lane * 16;
#pragma unroll
        for (int r = 0; r < 4; ++r) {
          u32x4 v;
#pragma unroll
          for (int e = 0; e < 4; ++e)
            v[e] = __builtin_bit_cast(unsigned, __builtin_convertvector((f32x2{acc[2 * e][m][r], acc[2 * e + 1][m][r]}), bf16x2));
          st128<AUX_SYS>(v, self, yo + r * 1024);
        }
      }
      wait_vm<0>();
      if (lane < 16 * nearly && (lane & 7) != rank) {
        glb_u64* fp = (glb_u64*)(P.ws[lane & 7] + FY_SRC + (rank * 256 + ((wave >> 1) * 4 + (lane >> 3)) * 16 + tj * 2 + (wave & 1)) * 8);
        __hip_atomic_store(fp, pair(seq, seq), __ATOMIC_RELAXED, __HIP_MEMORY_SCOPE_SYSTEM);
      }
    }
    if (wave == 0) {
      unsigned gm = 0, tm = 0;
      unsigned long long t0 = now();
      for (unsigned spins = 0;; ++spins) {
        asm volatile("" ::: "memory");
        u32x2 g = __builtin_amdgcn_raw_buffer_load_b64(self, AMAX_G + (b & 7) * 128 + (lane & 7) * 8, 0, AUX_SYS);
        tm = rfl(__builtin_amdgcn_raw_buffer_load_b32(self, TMO, 0, AUX_SYS));
        if (__all(g[1] >= seq)) { gm = g[0]; break; }
        if (tm == seq || ((spins & 63) == 63 && give_up(self, seq, t0))) { tm = seq; break; }
        __builtin_amdgcn_s_sleep(1);
      }
      const float amax = __uint_as_float(wave_max_u32(gm));
      float iv = amax > 0.f ? 448.f / amax : 0.f;
      if (tm == seq) iv = __uint_as_float(0x7fc00000u);
      if (lane == 0) {
        lds_st(smem, L_SCALE, __float_as_uint(iv));
        lds_st(smem, L_SFLAG, 1u);
      }
    }
    while (lds_ld(smem, L_SFLAG) == 0) __builtin_amdgcn_s_sleep(1);
    asm volatile("" ::: "memory");
    const float inv = __uint_as_float(lds_ld(smem, L_SCALE));
    if (wave != (YROWS / 64) * 2) __builtin_amdgcn_s_sleep(OUT_DELAY);
    char* tb = smem + wave * 8192;
    const unsigned c = lane & 15, q = lane >> 4;
    const unsigned ovoff = (lane >> 3) * NDIM + (lane & 7) * 16;
    const unsigned osoff = (ti * 256 + (wave >> 1) * 64) * NDIM + rank * NBLK + tj * 256 + (wave & 1) * 128;
    u32x4 ov[8];
#pragma unroll
    for (int hf = 0; hf < 2; ++hf) {
#pragma unroll
      for (int m = 2 * hf; m < 2 * hf + 2; ++m) {
        unsigned dw[8];
#pragma unroll
        for (int r = 0; r < 4; ++r)
#pragma unroll
          for (int hh = 0; hh < 2; ++hh) {
            int p = __builtin_amdgcn_cvt_pk_fp8_f32(acc[4 * hh + 0][m][r] * inv, acc[4 * hh + 1][m][r] * inv, 0, false);
            p = __builtin_amdgcn_cvt_pk_fp8_f32(acc[4 * hh + 2][m][r] * inv, acc[4 * hh + 3][m][r] * inv, p, true);
            dw[2 * r + hh] = (unsigned)p;
          }
        const unsigned row = m * 16 + c;
        *(u32x4*)(tb + row * 128 + (((2 * q) ^ (row & 7)) * 16)) = u32x4{dw[0], dw[1], dw[2], dw[3]};
        *(u32x4*)(tb + row * 128 + (((2 * q + 1) ^ (row & 7)) * 16)) = u32x4{dw[4], dw[5], dw[6], dw[7]};
      }
#pragma unroll
      for (int it = 4 * hf; it < 4 * hf + 4; ++it) {
        const unsigned row = it * 8 + (lane >> 3);
        ov[it] = *(const u32x4*)(tb + row * 128 + (((lane & 7) ^ (row & 7)) * 16));
      }
      const unsigned chunk = (((wave >> 1) * 2 + hf) * 2 + (wave & 1)) * 128 + b;
      if ((unsigned)hf >= nearly) {
#pragma unroll
        for (int it = 4 * hf; it < 4 * hf + 4; ++it)
          st128<AUX_SYS>(ov[it], self, EXP_O + chunk * 4096 + lane * 16 + (it & 3) * 1024);
      }
    }
    wait_vm<0>();
    if (lane < 2 * NR && (lane >> 3) >= nearly && (lane & 7) != rank) {
      glb_u64* fp = (glb_u64*)(P.ws[lane & 7] + FO_SRC + (rank * 2048 + (((wave >> 1) * 2 + (lane >> 3)) * 2 + (wave & 1)) * 128 + b) * 8);
      __hip_atomic_store(fp, pair(seq, seq), __ATOMIC_RELAXED, __HIP_MEMORY_SCOPE_SYSTEM);
    }
    __amdgpu_buffer_rsrc_t ro = mk_rsrc(P.out, OUT_BYTES);
#pragma unroll
    for (int it = 0; it < 8; ++it) st128<AUX_SC1>(ov[it], ro, osoff + ovoff + it * 8 * NDIM);
    for (unsigned job = b * 8 + wave; job < 7 * YROWS; job += 1024) {
      const unsigned p = (rank + 1 + job % 7) & 7, id = job / 7;
      const bool ok = wait_flag(self, FY_RDY + (p * 256 + id) * 8, seq);
      const float iv = ok ? inv : __uint_as_float(0x7fc00000u);
      unsigned dw[8];
#pragma unroll
      for (int r = 0; r < 4; ++r) {
        const u32x4 v = __builtin_amdgcn_raw_buffer_load_b128(self, STG_Y + (p * 256 + id) * 4096 + r * 1024 + lane * 16, 0, AUX_SYS);
#pragma unroll
        for (int hh = 0; hh < 2; ++hh) {
          int pk = __builtin_amdgcn_cvt_pk_fp8_f32(__uint_as_float(v[2 * hh] << 16) * iv, __uint_as_float(v[2 * hh] & 0xffff0000u) * iv, 0, false);
          pk = __builtin_amdgcn_cvt_pk_fp8_f32(__uint_as_float(v[2 * hh + 1] << 16) * iv, __uint_as_float(v[2 * hh + 1] & 0xffff0000u) * iv, pk, true);
          dw[2 * r + hh] = (unsigned)pk;
        }
      }
      *(u32x4*)(tb + c * 128 + (((2 * q) ^ (c & 7)) * 16)) = u32x4{dw[0], dw[1], dw[2], dw[3]};
      *(u32x4*)(tb + c * 128 + (((2 * q + 1) ^ (c & 7)) * 16)) = u32x4{dw[4], dw[5], dw[6], dw[7]};
#pragma unroll
      for (int it = 0; it < 2; ++it) {
        const unsigned row = it * 8 + (lane >> 3);
        const u32x4 o = *(const u32x4*)(tb + row * 128 + (((lane & 7) ^ (row & 7)) * 16));
        st128<AUX_SC1>(o, ro, ((id >> 4) * 16 + it * 8) * NDIM + p * NBLK + (id & 15) * 128 + ovoff);
      }
    }
  } else if (wave == 0 && b < NPULLERS) {
    const unsigned seq = rfl(__builtin_amdgcn_raw_buffer_load_b32(self, EPOCH, 0, AUX_SYS)) + 1;
    const unsigned pq = b % 7, k = b / 7;
    const unsigned p = (rank + 1 + pq) & 7;
    const unsigned rotx = 8 * ((pq * 15 + 3) / 7);
    const unsigned voff = lane * 16;
    const u32x4 dp = mk_desc(P.ws[p], WS_NEED), dout = mk_desc(P.out, OUT_BYTES);
    const unsigned ovoff = (lane >> 3) * NDIM + (lane & 7) * 16;
    const unsigned ovo[4] = {ovoff, ovoff + 8 * NDIM, ovoff + 16 * NDIM, ovoff + 24 * NDIM};
    const u32x2 seq2 = {seq, seq};
    const u32x4 nan4 = {0x7f7f7f7fu, 0x7f7f7f7fu, 0x7f7f7f7fu, 0x7f7f7f7fu};
    auto run = [&](auto phase_tag) {
      constexpr int PH = decltype(phase_tag)::value;
      constexpr unsigned J0 = PH == 0 ? 4 * PUSH_W : 0;
      constexpr unsigned NJ = PH == 0 ? 1536u + YROWS : 2048u - YROWS / 2;
      const unsigned tk_off = TICKET + (PH * 8 + pq) * 128;
      const unsigned tk_base = (seq - 1) * (NJ - J0 - 2 * NP) - 3 * NP - J0;
      auto decode = [&](unsigned n, unsigned& kind, unsigned& idx) {
        if (n < 72) { kind = n < 64; idx = n < 64 ? p * 512 + n : p * 1024 + n - 64; return; }
        if (n >= 1536) { kind = 2; idx = n - 1536; return; }
        const unsigned s = (n - 72) / 192, r = (n - 72) % 192;
        const unsigned t = r / 3, e = r % 3;
        if (s < 7 && e == 2) { kind = 1; idx = (p * 8 + s + 1) * 64 + ((t + 8 * (pq + 1)) & 63); return; }
        kind = 0;
        const unsigned xi = s < 7 ? t * 2 + e : r + 8;
        if (xi < 8) { idx = (p * 8 + s + 1) * 128 + xi; return; }
        unsigned cx = xi - 8 + rotx;
        if (cx >= 120) cx -= 120;
        idx = (p * 8 + s) * 128 + 8 + cx;
      };
      auto ochunk = [&](unsigned n) -> unsigned {
        constexpr unsigned G0 = YROWS / 16, FULL = (16 - G0) * 128;
        if (n < FULL) return G0 * 128 + n;
        const unsigned g = (n - FULL) / 120, i = (n - FULL) % 120;
        return g * 128 + (i < 24 ? (i / 6) * 8 + 2 + i % 6 : i + 8);
      };
      auto f_src = [&](unsigned n) -> unsigned {
        if constexpr (PH == 1) return FO_SRC + (p * 2048 + ochunk(n)) * 8;
        unsigned kind, idx;
        decode(n, kind, idx);
        return kind == 0 ? FX_SRC + idx * 8 : kind == 1 ? FW_SRC + (idx >> 2) * 8 : FY_SRC + (p * 256 + idx) * 8;
      };
      auto addrs = [&](unsigned n, unsigned& src, unsigned& dst, unsigned& frdy) {
        if constexpr (PH == 1) {
          const unsigned ch = ochunk(n);
          const unsigned tb = ch & 127, g = tb & 7, x = tb >> 3;
          const unsigned i = (g >> 1) * 4 + (x >> 2), j = (g & 1) * 4 + (x & 3);
          src = EXP_O + ch * 4096;
          dst = (i * 256 + (ch >> 8) * 32) * NDIM + p * NBLK + j * 256 + ((ch >> 7) & 1) * 128;
          frdy = 0;
        } else {
          unsigned kind, idx;
          decode(n, kind, idx);
          if (kind == 2) {
            src = EXP_Y + idx * 4096;
            dst = STG_Y + (p * 256 + idx) * 4096;
            frdy = FY_RDY + (p * 256 + idx) * 8;
          } else {
            dst = (kind ? STG_W : STG_X) + idx * 4096;
            src = kind ? EXP_W + (rank * 512 + (idx & 511)) * 4096 : dst;
            frdy = (kind ? FW_RDY : FX_RDY) + idx * 8;
          }
        }
      };
      u32x4 va[4], vb[4];
      unsigned j1 = J0 + k + NP, j2 = J0 + k + 2 * NP;
      unsigned src, dst, frdy, pfrdy;
      bool ok = wait_flag(self, f_src(J0 + k), seq);
      addrs(J0 + k, src, dst, frdy);
      u32x2 fv;
      unsigned dummy = 0;
      a_ld4(va, voff, dp, src);
      a_ld2(fv, ds, f_src(j1));
      unsigned nsrc, ndst, nfrdy;
      addrs(j1, nsrc, ndst, nfrdy);
      a_wait<0>(va, dummy, fv);
      auto step = [&](u32x4 (&cur)[4], u32x4 (&nxt)[4]) -> bool {
        if (j1 >= NJ) return false;
        const bool cur_ok = ok;
        if (rfl(min(fv[0], fv[1])) < seq) ok = wait_flag(self, f_src(j1), seq) && ok;
        a_ld4(nxt, voff, dp, nsrc);
        unsigned tk = 1;
        const bool has2 = j2 < NJ;
        if (has2) a_ticket_flag(tk, fv, ds, tk_off, f_src(j2));
        const unsigned cdst = dst;
        pfrdy = frdy; dst = ndst; frdy = nfrdy;
        if constexpr (PH == 0) {
          a_st4_sc1(cur, voff, ds, cdst);
        } else {
          if (!cur_ok) { cur[0] = nan4; cur[1] = nan4; cur[2] = nan4; cur[3] = nan4; }
          a_st4_rows(cur, ovo, dout, cdst);
        }
        j1 = j2;
        addrs(has2 ? j1 : J0, nsrc, ndst, nfrdy);
        asm volatile("" : "+s"(nsrc), "+s"(ndst), "+s"(nfrdy));
        if constexpr (PH == 0) {
          a_wait<0>(nxt, tk, fv);
          a_st2_sc1(seq2, ds, pfrdy);
        } else {
          a_wait<4>(nxt, tk, fv);
        }

        j2 = has2 ? rfl(tk) - tk_base : NJ;
        if (j2 > NJ) j2 = NJ;
        return true;
      };
      auto last = [&](u32x4 (&cur)[4]) {
        if constexpr (PH == 0) {
          a_st4_sc1(cur, voff, ds, dst);
          wait_vm<0>();
          a_st2_sc1(seq2, ds, frdy);
        } else {
          if (!ok) { cur[0] = nan4; cur[1] = nan4; cur[2] = nan4; cur[3] = nan4; }
          a_st4_rows(cur, ovo, dout, dst);
        }
      };
      for (;;) {
        if (!step(va, vb)) { last(va); break; }
        if (!step(vb, va)) { last(vb); break; }
      }
    };
    run(std::integral_constant<int, 0>{});
    run(std::integral_constant<int, 1>{});
  } else if (wave == 0 && b == REDUCER) {
    const unsigned seq = rfl(__builtin_amdgcn_raw_buffer_load_b32(self, EPOCH, 0, AUX_SYS)) + 1;
    unsigned gm = 0;
    unsigned long long t0 = now();
    for (unsigned spins = 0;; ++spins) {
      asm volatile("" ::: "memory");
      u32x4 a = __builtin_amdgcn_raw_buffer_load_b128(self, AMAX_WG + lane * 16, 0, AUX_SYS);
      gm = max(a[0], a[2]);
      if (__all(a[1] >= seq && a[3] >= seq)) break;
      if ((spins & 63) == 63 && give_up(self, seq, t0)) break;
    }
    gm = wave_max_u32(gm);
    glb_u64* gp = (glb_u64*)(P.ws[lane >> 3] + AMAX_G + (lane & 7) * 128 + rank * 8);
    __hip_atomic_store(gp, pair(gm, seq), __ATOMIC_RELAXED, __HIP_MEMORY_SCOPE_SYSTEM);
    if (lane == 0) __builtin_amdgcn_raw_buffer_store_b32(seq, self, EPOCH, 0, AUX_SYS);
  }
}

}

extern "C" void kernel_launch(int rank, int nranks, ncclComm_t comm, hipStream_t stream, void* const* d_in,
                              const int* in_sizes, int n_in, void* d_out, int out_size, const int* in_dtypes,
                              int out_dtype, void* const* d_out_all, void* const* d_ws_all, size_t ws_bytes) {
  (void)comm; (void)in_sizes; (void)in_dtypes; (void)out_dtype; (void)out_size; (void)d_out_all;
  if (nranks != NR || n_in < 2 || (unsigned)rank >= (unsigned)NR || ws_bytes < WS_NEED) {
    fprintf(stderr, "kernel_launch: unsupported configuration\n");
    return;
  }
  Params P;
  P.x = (const char*)d_in[0];
  P.w = (const char*)d_in[1];
  for (int p = 0; p < NR; ++p) P.ws[p] = (char*)d_ws_all[p];
  P.out = (char*)d_out;
  P.rank = rank;
  hipLaunchKernelGGL(fused_kernel, dim3(NWG), dim3(NTHREADS), 0, stream, P);
  hipError_t e = hipGetLastError();
  if (e != hipSuccess) fprintf(stderr, "launch failed: %s\n", hipGetErrorString(e));
}
```
